```python
import numpy as np

kernel_name = "gpu_gemm_rs_m4096_k4096_n4096_w8a8_dequant"


ABSMAX_RTOL = 0.001

COMPUTE_DTYPE = "int8"
EPILOGUE = "dequant"
IN_QUANT = "int8"
OUT_QUANT = None
IN_DTYPES = {"x": "int8", "w_mat": "int8",
             "scale_x": "float32", "scale_w": "float32"}
WORLD_SIZE = 8

def setup_inputs():

    x = np.random.randint(-127, 128, size=(4096, 4096)).astype(np.int8)
    w_mat = np.random.randint(-127, 128, size=(4096, 4096)).astype(np.int8)
    scale_x = (0.02 * (1.0 + np.random.rand(1))).astype(np.float32)
    scale_w = (0.02 * (1.0 + np.random.rand(1))).astype(np.float32)
    return {"x": x, "w_mat": w_mat, "scale_x": scale_x, "scale_w": scale_w}

def reference(x, w_mat, scale_x, scale_w):
    acc = x.astype(np.float64) @ w_mat.astype(np.float64)
    y = (acc.astype(np.float32) * (scale_x * scale_w)).astype(np.float32)
    return y
```

```cpp
#include <hip/hip_runtime.h>
#include <rccl/rccl.h>
#include <cstdio>
#include <cstdint>

#define HIP_CHECK(expr) do { hipError_t _e = (expr); if (_e != hipSuccess) { \
    fprintf(stderr, "HIP error %s at %s:%d\n", hipGetErrorString(_e), __FILE__, __LINE__); \
    return; } } while (0)

namespace {

typedef unsigned int u32;
typedef unsigned long long u64;
typedef __attribute__((ext_vector_type(4))) int i32x4;
typedef __attribute__((ext_vector_type(4))) float f32x4;

constexpr int NRANK = 8;
constexpr int GM = 4096, GN = 4096, GK = 4096;
constexpr int MS = GM / NRANK;
constexpr int KS = GK / NRANK;
constexpr int NWG = 256;
constexpr int NTHREADS = 576;
constexpr int NPHASE = 2;
constexpr int NGRP = 32;
constexpr int NCH = 8;

constexpr size_t XS_SRC = (size_t)MS * KS;
constexpr size_t WS_SRC = (size_t)KS * GN;
constexpr size_t XS_OFF = 0;
constexpr size_t WSB_OFF = XS_SRC * NRANK;
constexpr size_t PAR_BYTES = (XS_SRC + WS_SRC) * NRANK;
constexpr size_t CTRL_OFF = 2 * PAR_BYTES;
constexpr size_t FLAG_OFF = CTRL_OFF + 4096;
constexpr size_t WS_NEEDED = FLAG_OFF + (size_t)NRANK * NGRP * NCH * 4;
constexpr u64 DEADLINE_TICKS = 20000000ull;

struct Params {
  char* ws[NRANK];
  const int8_t* x;
  const int8_t* w;
  const float* sx;
  const float* sw;
  float* out;
  int rank;
};

__device__ __forceinline__ void store_wt(void* p, i32x4 v) {
  asm volatile("global_store_dwordx4 %0, %1, off sc0 sc1\n\ts_nop 1" ::"v"(p), "v"(v) : "memory");
}
template <int N> __device__ __forceinline__ void wait_vm() {
  asm volatile("s_waitcnt vmcnt(%0)" ::"n"(N) : "memory");
}

__device__ __forceinline__ void transpose4(u32 w0, u32 w1, u32 w2, u32 w3, u32& o0, u32& o1, u32& o2, u32& o3) {
  u32 t0 = __builtin_amdgcn_perm(w1, w0, 0x05010400u);
  u32 t1 = __builtin_amdgcn_perm(w1, w0, 0x07030602u);
  u32 t2 = __builtin_amdgcn_perm(w3, w2, 0x05010400u);
  u32 t3 = __builtin_amdgcn_perm(w3, w2, 0x07030602u);
  o0 = __builtin_amdgcn_perm(t2, t0, 0x05040100u);
  o1 = __builtin_amdgcn_perm(t2, t0, 0x07060302u);
  o2 = __builtin_amdgcn_perm(t3, t1, 0x05040100u);
  o3 = __builtin_amdgcn_perm(t3, t1, 0x07060302u);
}

struct Smem {
  i32x4 part[8][16][64];
  u32 call;
  u32 arrive;
  u32 reduced;
  u32 bad;
};

#define LDS_LOAD(p) __hip_atomic_load((p), __ATOMIC_RELAXED, __HIP_MEMORY_SCOPE_WORKGROUP)

__device__ __forceinline__ u32 poll_ready(u32* fptr, u32 fval, int nlanes, u32 todo, int lane, u32* tmo, u32 call) {
  u64 t0 = 0;
  for (u32 spins = 0;; ++spins) {
    u32 f = fval;
    if (lane < nlanes) f = __hip_atomic_load(fptr, __ATOMIC_RELAXED, __HIP_MEMORY_SCOPE_SYSTEM);
    const u64 b = __ballot((int)(f - fval) >= 0);
    const u32 ready = (u32)(b & (b >> 8) & (b >> 16) & (b >> 24) & (b >> 32)) & todo;
    if (ready) { asm volatile("" ::: "memory"); return ready; }
    if (spins == 0) t0 = __builtin_amdgcn_s_memrealtime();
    if ((spins & 63) == 63) {
      if (__hip_atomic_load(tmo, __ATOMIC_RELAXED, __HIP_MEMORY_SCOPE_AGENT) == call ||
          __builtin_amdgcn_s_memrealtime() - t0 > DEADLINE_TICKS) {
        __hip_atomic_store(tmo, call, __ATOMIC_RELAXED, __HIP_MEMORY_SCOPE_AGENT);
        return 0;
      }
    }
    __builtin_amdgcn_s_sleep(4);
  }
}

__global__ void __launch_bounds__(NTHREADS) gemm_gather_kernel(Params P) {
  __shared__ __attribute__((aligned(16))) Smem sm;
  const int tid = threadIdx.x;
  const int lane = tid & 63;
  const int wave = __builtin_amdgcn_readfirstlane(tid >> 6);
  const int bid = blockIdx.x;
  const int rank = P.rank;
  char* myws = P.ws[rank];
  u32* ctr = (u32*)(myws + CTRL_OFF);
  u32* tmo = (u32*)(myws + CTRL_OFF + 256);

  if (tid == 0) { sm.call = 0; sm.arrive = 0; sm.reduced = 0; sm.bad = 0; }
  __syncthreads();

  const int g = (bid & 7) * 4 + ((bid >> 3) & 3);
  const int hi = bid >> 5;

  if (wave == 8) {
    const int c = hi;
    const u32 cnt = __hip_atomic_load(ctr, __ATOMIC_RELAXED, __HIP_MEMORY_SCOPE_AGENT);
    const int s = lane & 15, kg = lane >> 4;
    u32 raw[NPHASE][16];
    i32x4 xf[NRANK];
    const int8_t* wsrc = P.w + (size_t)(c * 64 + kg * 16) * GN + g * 64 + s * 4;
#pragma unroll
    for (int i = 0; i < 16; ++i) raw[0][i] = *(const u32*)(wsrc + (size_t)i * GN);
#pragma unroll
    for (int i = 0; i < NRANK; ++i) {
      const int d = (rank + c + i) & 7;
      xf[i] = *(const i32x4*)(P.x + (size_t)(d * MS + g * 16 + s) * KS + c * 64 + kg * 16);
    }
#pragma unroll
    for (int i = 0; i < 16; ++i) raw[1][i] = *(const u32*)(wsrc + (size_t)i * GN + GN / 2);

    const u32 call = __builtin_amdgcn_readfirstlane(cnt) / NWG + 1;
    const size_t par_off = (call & 1) ? PAR_BYTES : 0;
    const size_t x_off = par_off + XS_OFF + (size_t)rank * XS_SRC + (size_t)(c * 32 + g) * 1024 + lane * 16;
    const size_t f_off = FLAG_OFF + (size_t)((rank * NGRP + g) * NCH + c) * 4;

    i32x4 wf[NPHASE][4];
#pragma unroll
    for (int t = 0; t < NPHASE; ++t) {
#pragma unroll
      for (int q = 0; q < 4; ++q) {
        u32 o0, o1, o2, o3;
        transpose4(raw[t][4 * q], raw[t][4 * q + 1], raw[t][4 * q + 2], raw[t][4 * q + 3], o0, o1, o2, o3);
        wf[t][0][q] = (int)o0; wf[t][1][q] = (int)o1; wf[t][2][q] = (int)o2; wf[t][3][q] = (int)o3;
      }
    }
    asm volatile("s_waitcnt vmcnt(0)" ::: "memory");
#pragma unroll
    for (int i = 0; i < NRANK; ++i) asm volatile("" : "+v"(xf[i]));

#pragma unroll
    for (int t = 0; t < NPHASE; ++t) {
      const size_t w_off = par_off + WSB_OFF + (size_t)rank * WS_SRC +
                           (size_t)(((t * NGRP + g) * NCH + c) * 4) * 1024 + lane * 16;
      const u32 fval = 2 * call - 1 + t;
#pragma unroll
      for (int i = 0; i < NRANK; ++i) {
        const int d = (rank + c + i) & 7;
        char* dst = P.ws[d];
        if (t == 0) store_wt(dst + x_off, xf[i]);
#pragma unroll
        for (int j = 0; j < 4; ++j) store_wt(dst + w_off + j * 1024, wf[t][j]);
        wait_vm<0>();
        if (lane == 0) __hip_atomic_store((u32*)(dst + f_off), fval, __ATOMIC_RELAXED, __HIP_MEMORY_SCOPE_SYSTEM);
      }
      if (t == 0) {
        while (LDS_LOAD(&sm.call) == 0) __builtin_amdgcn_s_sleep(1);
        if (lane == 0) __hip_atomic_fetch_add(ctr, 1u, __ATOMIC_RELAXED, __HIP_MEMORY_SCOPE_AGENT);
      }
    }
    return;
  }

  const int m = hi;
  const int src = (rank - wave) & 7;
  u32 call;
  if (wave == 0) {
    const u32 cnt = __hip_atomic_load(ctr, __ATOMIC_RELAXED, __HIP_MEMORY_SCOPE_AGENT);
    call = __builtin_amdgcn_readfirstlane(cnt) / NWG + 1;
    __builtin_amdgcn_fence(__ATOMIC_ACQUIRE, "");
    asm volatile("s_waitcnt vmcnt(0)" ::: "memory");
    if (lane == 0) __hip_atomic_store(&sm.call, call, __ATOMIC_RELAXED, __HIP_MEMORY_SCOPE_WORKGROUP);
  } else {
    while ((call = LDS_LOAD(&sm.call)) == 0) __builtin_amdgcn_s_sleep(2);
    call = __builtin_amdgcn_readfirstlane(call);
  }
  const float scale = P.sx[0] * P.sw[0];
  const size_t par_off = (call & 1) ? PAR_BYTES : 0;
  const char* xbase = myws + par_off + XS_OFF + (size_t)src * XS_SRC + (size_t)(m * 4) * 1024 + lane * 16;
  const char* wbase = myws + par_off + WSB_OFF + (size_t)src * WS_SRC + (size_t)(g * NCH * 4) * 1024 + lane * 16;
  const int fg = lane < 8 ? g : (m * 4 + (((lane - 8) >> 3) & 3));
  u32* fptr = (u32*)(myws + FLAG_OFF) + (src * NGRP + fg) * NCH + (lane & 7);

#pragma unroll 1
  for (int t = 0; t < NPHASE; ++t) {
    i32x4 acc[4][4];
#pragma unroll
    for (int a = 0; a < 4; ++a)
#pragma unroll
      for (int b = 0; b < 4; ++b) acc[a][b] = i32x4{0, 0, 0, 0};
    const u32 fval = 2 * call - 1 + t;
    const char* wt = wbase + (size_t)t * (NGRP * NCH * 4 * 1024);
    u32 todo = 0xFF, ready = 0;
    i32x4 af[2][4], bf[2][4];
#pragma unroll
    for (int n = 0; n <= NCH; ++n) {
      if (n < NCH) {
        if (!ready) {
          ready = poll_ready(fptr, fval, t == 0 ? 40 : 8, todo, lane, tmo, call);
          if (!ready) {
            __hip_atomic_store(&sm.bad, 1u, __ATOMIC_RELAXED, __HIP_MEMORY_SCOPE_WORKGROUP);
            ready = todo;
          }
        }
        const int c = __builtin_ctz(ready);
        ready &= ready - 1;
        todo &= ~(1u << c);
        const char* xp = xbase + (size_t)c * (32 * 1024);
        const char* wp = wt + (size_t)c * 4096;
#pragma unroll
        for (int j = 0; j < 4; ++j) bf[n & 1][j] = *(const i32x4*)(wp + j * 1024);
#pragma unroll
        for (int a = 0; a < 4; ++a) af[n & 1][a] = *(const i32x4*)(xp + a * 1024);
        __builtin_amdgcn_sched_barrier(0);
      }
      if (n > 0) {
#pragma unroll
        for (int a = 0; a < 4; ++a)
#pragma unroll
          for (int j = 0; j < 4; ++j)
            acc[a][j] = __builtin_amdgcn_mfma_i32_16x16x64_i8(af[(n - 1) & 1][a], bf[(n - 1) & 1][j], acc[a][j], 0, 0, 0);
        __builtin_amdgcn_sched_barrier(0);
      }
    }
    if (t > 0) {
      while (LDS_LOAD(&sm.reduced) < 8u * t) __builtin_amdgcn_s_sleep(1);
    }
#pragma unroll
    for (int a = 0; a < 4; ++a)
#pragma unroll
      for (int i = 0; i < 4; ++i)
        sm.part[wave][a * 4 + i][lane] = i32x4{acc[a][0][i], acc[a][1][i], acc[a][2][i], acc[a][3][i]};
    asm volatile("s_waitcnt lgkmcnt(0)" ::: "memory");
    if (lane == 0) __hip_atomic_fetch_add(&sm.arrive, 1u, __ATOMIC_RELAXED, __HIP_MEMORY_SCOPE_WORKGROUP);
    while (LDS_LOAD(&sm.arrive) < 8u * (t + 1)) __builtin_amdgcn_s_sleep(1);
    asm volatile("" ::: "memory");
    const bool bad = LDS_LOAD(&sm.bad) != 0;
#pragma unroll
    for (int h = 0; h < 2; ++h) {
      const int e = wave * 2 + h;
      i32x4 sum = sm.part[0][e][lane];
#pragma unroll
      for (int w = 1; w < 8; ++w) sum += sm.part[w][e][lane];
      f32x4 y;
#pragma unroll
      for (int j = 0; j < 4; ++j) y[j] = bad ? __builtin_nanf("") : (float)sum[j] * scale;
      const int row = m * 64 + (e >> 2) * 16 + (lane >> 4) * 4 + (e & 3);
      const int col = t * (GN / 2) + g * 64 + (lane & 15) * 4;
      *(f32x4*)(P.out + (size_t)row * GN + col) = y;
    }
    asm volatile("s_waitcnt lgkmcnt(0)" ::: "memory");
    if (lane == 0) __hip_atomic_fetch_add(&sm.reduced, 1u, __ATOMIC_RELAXED, __HIP_MEMORY_SCOPE_WORKGROUP);
  }
}

}

extern "C" void kernel_launch(int rank, int nranks, ncclComm_t comm,
                              hipStream_t stream,
                              void* const* d_in, const int* in_sizes, int n_in,
                              void* d_out, int out_size,
                              const int* in_dtypes, int out_dtype,
                              void* const* d_out_all,
                              void* const* d_ws_all, size_t ws_bytes) {
  if (nranks != NRANK || n_in != 4 || ws_bytes < WS_NEEDED || out_size != MS * GN ||
      in_sizes[0] != GM * KS || in_sizes[1] != KS * GN || (unsigned)rank >= (unsigned)NRANK) {
    fprintf(stderr, "kernel_launch: unexpected problem shape\n");
    return;
  }
  Params p;
  for (int i = 0; i < NRANK; ++i) p.ws[i] = (char*)d_ws_all[i];
  p.x = (const int8_t*)d_in[0];
  p.w = (const int8_t*)d_in[1];
  p.sx = (const float*)d_in[2];
  p.sw = (const float*)d_in[3];
  p.out = (float*)d_out;
  p.rank = rank;
  hipLaunchKernelGGL(gemm_gather_kernel, dim3(NWG), dim3(NTHREADS), 0, stream, p);
  HIP_CHECK(hipGetLastError());
}
```

```cpp
#include <hip/hip_runtime.h>
#include <rccl/rccl.h>
#include <cstdio>
#include <cstdint>

#define HIP_CHECK(expr) do { hipError_t _e = (expr); if (_e != hipSuccess) { \
    fprintf(stderr, "HIP error %s at %s:%d\n", hipGetErrorString(_e), __FILE__, __LINE__); \
    return; } } while (0)

namespace {

typedef unsigned int u32;
typedef unsigned long long u64;
typedef __attribute__((ext_vector_type(4))) int i32x4;
typedef __attribute__((ext_vector_type(4))) unsigned u32x4;
typedef __attribute__((ext_vector_type(4))) float f32x4;

#define PWAVES 4
#define POLL_SLEEP 4
#define OUT_AUX 16
#define LD_AUX 16

constexpr int NRANK = 8;
constexpr int GM = 4096, GN = 4096, GK = 4096;
constexpr int MS = GM / NRANK;
constexpr int KS = GK / NRANK;
constexpr int NTHREADS = 512;
constexpr int NPUSH = 56;
constexpr int NEARLY = 0;
constexpr int NCONS = 192;
constexpr int NWG = NPUSH + NCONS;
constexpr int NSLOT = 8 * PWAVES;
constexpr int NBLK = 576;
constexpr int NCH = 8;
constexpr int AUX_SYS = 17;
constexpr int NROUND = NBLK / NSLOT;
static_assert(PWAVES == 4, "pusher geometry");
#define DYN 1
#define WATCH_SLEEP 20
constexpr u32 TICKETS_PER_CALL = NBLK - 6 * NSLOT + NSLOT;

constexpr size_t SRC_BYTES = (size_t)NBLK * 4096;
constexpr size_t PAR_BYTES = SRC_BYTES * NRANK;
constexpr size_t CTRL_OFF = 2 * PAR_BYTES;
constexpr size_t FLAG_OFF = CTRL_OFF + 4096;
constexpr size_t WS_NEEDED = FLAG_OFF + (size_t)NRANK * 72 * 128;
constexpr u64 DEADLINE_TICKS = 20000000ull;

struct Params {
  char* ws[NRANK];
  const int8_t* x;
  const int8_t* w;
  const float* sx;
  const float* sw;
  float* out;
  int rank;
};

#define STAMP(slot) do {} while (0)

#define WAIT_VM0() do { asm volatile("" ::: "memory"); __builtin_amdgcn_s_waitcnt(0x0F70); asm volatile("" ::: "memory"); } while (0)
#define WAIT_VMN(n) do { asm volatile("" ::: "memory"); __builtin_amdgcn_s_waitcnt(0x0F70 | (n)); asm volatile("" ::: "memory"); } while (0)
#define FENCE() asm volatile("" ::: "memory")
#define LDS_LOAD(p) __hip_atomic_load((p), __ATOMIC_RELAXED, __HIP_MEMORY_SCOPE_WORKGROUP)
#define LDS_STORE(p, v) __hip_atomic_store((p), (v), __ATOMIC_RELAXED, __HIP_MEMORY_SCOPE_WORKGROUP)

__device__ __forceinline__ void transpose4(u32 w0, u32 w1, u32 w2, u32 w3, u32& o0, u32& o1, u32& o2, u32& o3) {
  u32 t0 = __builtin_amdgcn_perm(w1, w0, 0x05010400u);
  u32 t1 = __builtin_amdgcn_perm(w1, w0, 0x07030602u);
  u32 t2 = __builtin_amdgcn_perm(w3, w2, 0x05010400u);
  u32 t3 = __builtin_amdgcn_perm(w3, w2, 0x07030602u);
  o0 = __builtin_amdgcn_perm(t2, t0, 0x05040100u);
  o1 = __builtin_amdgcn_perm(t2, t0, 0x07060302u);
  o2 = __builtin_amdgcn_perm(t3, t1, 0x05040100u);
  o3 = __builtin_amdgcn_perm(t3, t1, 0x07060302u);
}

__device__ __forceinline__ void decode_block(int n, int& row, int& c) {
  if (n < 64) { row = 64 + (n & 7); c = n >> 3; return; }
  const int i = n - 64;
  if (i < 384) { const int ph = i >= 192, j = i - ph * 192; c = j / 24; row = ph * 24 + j - c * 24; }
  else { const int j = i - 384; c = j >> 4; row = 48 + (j & 15); }
}
__device__ __forceinline__ void w_index(int grp, int& first, int& step) {
  if (grp < 24) { first = 64 + grp; step = 24; }
  else if (grp < 48) { first = 64 + 192 + grp - 24; step = 24; }
  else { first = 64 + 384 + grp - 48; step = 16; }
}

template <class R>
__device__ __forceinline__ void load_x_block(R rx, int dest, int n, int lane, u32x4 (&o)[4]) {
  const int soff = (dest * MS + (n & 7) * 64) * KS + (n >> 3) * 64;
#pragma unroll
  for (int a = 0; a < 4; ++a)
    o[a] = __builtin_amdgcn_raw_buffer_load_b128(rx, (lane & 15) * KS + (lane >> 4) * 16, soff + a * 16 * KS, 0);
}

template <class R>
__device__ __forceinline__ void load_w_block(R rw, int grp, int c, int lane, u32 (&raw)[16]) {
#pragma unroll
  for (int i = 0; i < 16; ++i)
    raw[i] = __builtin_amdgcn_raw_buffer_load_b32(rw, (lane >> 4) * 16 * GN + (lane & 15) * 4, (c * 64 + i) * GN + grp * 64, 0);
}
__device__ __forceinline__ void tile_w_block(const u32 (&raw)[16], u32x4 (&o)[4]) {
#pragma unroll
  for (int k = 0; k < 4; ++k) {
    u32 o0, o1, o2, o3;
    transpose4(raw[4 * k], raw[4 * k + 1], raw[4 * k + 2], raw[4 * k + 3], o0, o1, o2, o3);
    o[0][k] = o0; o[1][k] = o1; o[2][k] = o2; o[3][k] = o3;
  }
}

struct Smem {
  union {
    u64 tile[3][16][2][64];
    char fill[96 * 1024];
  };
  u32 arrive[4];
  u32 call;
  u32 bad;
  u32 ready;
  u32 pad;
};

__device__ __forceinline__ bool deadline_hit(u32 spins, u64& t0, u32* tmo, u32 call) {
  if (spins == 0) t0 = __builtin_amdgcn_s_memrealtime();
  if ((spins & 63) != 63) return false;
  if (__hip_atomic_load(tmo, __ATOMIC_RELAXED, __HIP_MEMORY_SCOPE_AGENT) == call ||
      __builtin_amdgcn_s_memrealtime() - t0 > DEADLINE_TICKS) {
    __hip_atomic_store(tmo, call, __ATOMIC_RELAXED, __HIP_MEMORY_SCOPE_AGENT);
    return true;
  }
  return false;
}

__device__ __forceinline__ u32 poll_ready(u32* fptr, int nlanes, u32 todo, int lane, u32* tmo, u32 call) {
  u64 t0 = 0;
  for (u32 spins = 0;; ++spins) {
    u32 f = call;
    if (lane < nlanes) f = __hip_atomic_load(fptr, __ATOMIC_RELAXED, __HIP_MEMORY_SCOPE_SYSTEM);
    const u64 b = __ballot((int)(f - call) >= 0);
    const u32 ready = (u32)(b & (b >> 8)) & todo;
    if (ready) { FENCE(); return ready; }
    if (deadline_hit(spins, t0, tmo, call)) return 0;
    __builtin_amdgcn_s_sleep(POLL_SLEEP);
  }
}

__global__ void __launch_bounds__(NTHREADS) gemm_gather_kernel(Params P) {
  __shared__ __attribute__((aligned(16))) Smem sm;
  const int tid = threadIdx.x;
  const int lane = tid & 63;
  const int wave = __builtin_amdgcn_readfirstlane(tid >> 6);
  const int bid = blockIdx.x;
  const int rank = P.rank;
  char* myws = P.ws[rank];
  u32* ctr = (u32*)(myws + CTRL_OFF);
  u32* tmo = (u32*)(myws + CTRL_OFF + 256);
  u32* tiled = (u32*)(myws + CTRL_OFF + 512);
  const auto rx = __builtin_amdgcn_make_buffer_rsrc((void*)P.x, 0, GM * KS, 0x00020000);
  const auto rl = __builtin_amdgcn_make_buffer_rsrc(myws, 0, (int)WS_NEEDED, 0x00020000);
  const auto rw = __builtin_amdgcn_make_buffer_rsrc((void*)P.w, 0, KS * GN, 0x00020000);

  if (bid < NPUSH) {
    if (tid == 0) sm.ready = 0;
    if (wave < PWAVES) {
      const int dest = (rank + 1 + (bid >> 3)) & 7;
      const int slot = (bid & 7) * PWAVES + wave;
      u32x4 buf[2][4], own[2][4];
      u32 raw[2][16];
      load_x_block(rx, dest, slot, lane, buf[0]);
      const u32 cnt = __hip_atomic_load(ctr, __ATOMIC_RELAXED, __HIP_MEMORY_SCOPE_AGENT);
      load_x_block(rx, dest, NSLOT + slot, lane, buf[1]);
      FENCE();
      const u32 call = (u32)__builtin_amdgcn_readfirstlane(cnt) / (u32)NWG + 1;
      const auto rs = __builtin_amdgcn_make_buffer_rsrc(P.ws[dest], 0, (int)WS_NEEDED, 0x00020000);
      const u32 dbase = (u32)((call & 1) ? PAR_BYTES : 0) + (u32)rank * (u32)SRC_BYTES;
      const u32 fbase = (u32)FLAG_OFF + (u32)rank * (72 * 128);
      const int lane0 = lane == 0 ? 0 : 0x7FFFFFF0;
      auto send = [&](u32 n, const u32x4 (&d)[4]) __attribute__((always_inline)) {
#pragma unroll
        for (int j = 0; j < 4; ++j)
          __builtin_amdgcn_raw_buffer_store_b128(d[j], rs, lane * 16 + j * 1024, dbase + n * 4096, AUX_SYS);
        FENCE();
      };
      auto fetch = [&](u32 n, u32x4 (&d)[4]) __attribute__((always_inline)) {
#pragma unroll
        for (int j = 0; j < 4; ++j)
          d[j] = __builtin_amdgcn_raw_buffer_load_b128(rl, lane * 16 + j * 1024, dbase + n * 4096, LD_AUX);
        FENCE();
      };
      auto flag = [&](u32 n) __attribute__((always_inline)) {
        int fr, fc;
        decode_block((int)n, fr, fc);
        const u32 foff = __builtin_amdgcn_readfirstlane(fbase + (u32)fr * 128 + (u32)fc * 4);
        __builtin_amdgcn_raw_buffer_store_b32(call, rs, lane0, foff, AUX_SYS);
        FENCE();
      };
      send(slot, buf[0]);
#pragma unroll
      for (int h = 0; h < 2; ++h) {
        int row, c;
        decode_block((2 + h) * NSLOT + slot, row, c);
        load_w_block(rw, row, c, lane, raw[h]);
      }
      asm volatile("s_waitcnt lgkmcnt(0)" ::: "memory");
      __builtin_amdgcn_s_barrier();
      WAIT_VM0(); flag(slot); send(NSLOT + slot, buf[1]);
      tile_w_block(raw[0], own[0]);
      tile_w_block(raw[1], own[1]);
#pragma unroll
      for (int j = 0; j < 4; ++j) asm volatile("" : "+v"(own[0][j]), "+v"(own[1][j]));
      WAIT_VM0(); flag(NSLOT + slot); send(2 * NSLOT + slot, own[0]);
      WAIT_VM0();
      if (LDS_LOAD(&sm.ready) != 0) { FENCE(); fetch(4 * NSLOT + slot, buf[0]); flag(2 * NSLOT + slot); send(3 * NSLOT + slot, own[1]); }
      else {
        flag(2 * NSLOT + slot); send(3 * NSLOT + slot, own[1]);
        while (LDS_LOAD(&sm.ready) == 0) __builtin_amdgcn_s_sleep(1);
        FENCE();
        fetch(4 * NSLOT + slot, buf[0]);
      }
      u32 prv = 3 * NSLOT + slot, cur = 4 * NSLOT + slot, nxt = 5 * NSLOT + slot;
      const int qoff = (int)CTRL_OFF + 2048 + (bid >> 3) * 128;
      int vzero = 0;
      asm volatile("" : "+v"(vzero));
      const u32 qbase = (call - 1) * TICKETS_PER_CALL - 6 * NSLOT;
      auto round = [&](u32x4 (&dc)[4], u32x4 (&dn)[4]) __attribute__((always_inline)) {
        u32 tk = 0;
        if (nxt < (u32)NBLK) {
          fetch(nxt, dn);
          if (lane == 0) tk = (u32)__builtin_amdgcn_raw_ptr_buffer_atomic_add_i32(1, rl, vzero, qoff, 0);
          FENCE();
        }
        flag(prv);
        send(cur, dc);
        WAIT_VM0();
        prv = cur; cur = nxt;
        nxt = nxt < (u32)NBLK ? (u32)__builtin_amdgcn_readfirstlane(tk) - qbase : (u32)NBLK;
      };
      WAIT_VM0();
#pragma unroll 1
      for (;;) {
        round(buf[0], buf[1]);
        if (cur >= (u32)NBLK) break;
        round(buf[1], buf[0]);
        if (cur >= (u32)NBLK) break;
      }
      if (LDS_LOAD(&sm.ready) == 1) flag(prv);
      STAMP(2);
    } else {
      const u32 cnt = __hip_atomic_load(ctr, __ATOMIC_RELAXED, __HIP_MEMORY_SCOPE_AGENT);
      __builtin_amdgcn_s_barrier();
      const u32 call = (u32)__builtin_amdgcn_readfirstlane(cnt) / (u32)NWG + 1;
      const u32 target = call * (NCONS / 8);
      u64 t0 = 0;
      for (int k = PWAVES; k <= wave; ++k) __builtin_amdgcn_s_sleep(WATCH_SLEEP);
      for (u32 spins = 0; LDS_LOAD(&sm.ready) == 0; ++spins) {
        u32 f = target;
        if (lane < 8) f = __hip_atomic_load(tiled + lane * 32, __ATOMIC_RELAXED, __HIP_MEMORY_SCOPE_AGENT);
        if (__all((int)(f - target) >= 0)) { if (lane == 0) LDS_STORE(&sm.ready, 1u); break; }
        if (deadline_hit(spins, t0, tmo, call)) { if (lane == 0) LDS_STORE(&sm.ready, 2u); break; }
        __builtin_amdgcn_s_sleep(8);
      }
    }
    __syncthreads();
    if (tid == 0) __hip_atomic_fetch_add(ctr, 1u, __ATOMIC_RELAXED, __HIP_MEMORY_SCOPE_AGENT);
    return;
  }

  const int q = bid - NPUSH - NEARLY;
  const int xcd = bid & 7;
  const int m = (q >> 3) & 7;
  const int gl = q >> 6;
  const int src = (rank - wave) & 7;
  auto init_lds = [&]() __attribute__((always_inline)) {
    u64* z = &sm.tile[0][0][0][0];
#pragma unroll
    for (int i = 0; i < 3 * 16 * 2 * 64 / NTHREADS; ++i) z[i * NTHREADS + tid] = 0;
    if (tid < 4) sm.arrive[tid] = 0;
    if (tid == 4) sm.bad = 0;
    asm volatile("s_waitcnt lgkmcnt(0)" ::: "memory");
    __builtin_amdgcn_s_barrier();
  };
  u32 call;
  if (wave >= 5) {
    const int n = (128 + (wave - 5) * NCONS + q) % NBLK;
    int row, c;
    decode_block(n, row, c);
    u32x4 o[4];
    u32 raw[16];
    if (n < 64) load_x_block(rx, rank, n, lane, o);
    else load_w_block(rw, row, c, lane, raw);
    const u32 cnt = __hip_atomic_load(ctr, __ATOMIC_RELAXED, __HIP_MEMORY_SCOPE_AGENT);
    FENCE();
    init_lds();
    call = (u32)__builtin_amdgcn_readfirstlane(cnt) / (u32)NWG + 1;
    if (n >= 64) tile_w_block(raw, o);
    const u32 dbase = (u32)((call & 1) ? PAR_BYTES : 0) + (u32)rank * (u32)SRC_BYTES;
#pragma unroll
    for (int j = 0; j < 4; ++j)
      __builtin_amdgcn_raw_buffer_store_b128(o[j], rl, lane * 16 + j * 1024, dbase + (u32)n * 4096, AUX_SYS);
    WAIT_VM0();
    if (lane == 0) {
      __builtin_amdgcn_raw_buffer_store_b32(call, rl, 0, (u32)FLAG_OFF + (u32)rank * (72 * 128) + (u32)row * 128 + (u32)c * 4, AUX_SYS);
      if (__hip_atomic_fetch_add(&sm.arrive[3], 1u, __ATOMIC_RELAXED, __HIP_MEMORY_SCOPE_WORKGROUP) == 2)
        __hip_atomic_fetch_add(tiled + xcd * 32, 1u, __ATOMIC_RELAXED, __HIP_MEMORY_SCOPE_AGENT);
    }
    STAMP(36);
  } else {
    const u32 cnt = __hip_atomic_load(ctr, __ATOMIC_RELAXED, __HIP_MEMORY_SCOPE_AGENT);
    FENCE();
    init_lds();
    call = (u32)__builtin_amdgcn_readfirstlane(cnt) / (u32)NWG + 1;
  }
  const float scale = P.sx[0] * P.sw[0];
  const u32 sbase = (u32)((call & 1) ? PAR_BYTES : 0) + (u32)src * (u32)SRC_BYTES;
  u32* fsrc = (u32*)(myws + FLAG_OFF + (size_t)src * (72 * 128));
  const int nphase = gl < 2 ? 3 : 2;

#pragma unroll 1
  for (int t = 0; t < nphase; ++t) {
    const int grp = t < 2 ? t * 24 + xcd * 3 + gl : 48 + xcd * 2 + gl;
    i32x4 acc[4][4];
#pragma unroll
    for (int a = 0; a < 4; ++a)
#pragma unroll
      for (int b = 0; b < 4; ++b) acc[a][b] = i32x4{0, 0, 0, 0};
    int wfirst, wstep;
    w_index(grp, wfirst, wstep);
    u32* fptr = fsrc + (lane < 8 ? grp : 64 + m) * 32 + (lane & 7);
    u32 todo = 0xFF, ready = 0;
    u32x4 af[2][4], bf[2][4];
#pragma unroll
    for (int n = 0; n <= NCH; ++n) {
      if (n < NCH) {
        if (!ready) {
          ready = poll_ready(fptr, t == 0 ? 16 : 8, todo, lane, tmo, call);
          if (!ready) {
            LDS_STORE(&sm.bad, 1u);
            ready = todo;
          }
        }
        const int c = __builtin_ctz(ready);
        ready &= ready - 1;
        todo &= ~(1u << c);
        STAMP(t * 12 + n);
        const u32 xo = sbase + (u32)(c * 8 + m) * 4096;
        const u32 wo = __builtin_amdgcn_readfirstlane(sbase + (u32)(wfirst + c * wstep) * 4096);
#pragma unroll
        for (int j = 0; j < 4; ++j) bf[n & 1][j] = __builtin_amdgcn_raw_buffer_load_b128(rl, lane * 16 + j * 1024, wo, LD_AUX);
#pragma unroll
        for (int a = 0; a < 4; ++a) af[n & 1][a] = __builtin_amdgcn_raw_buffer_load_b128(rl, lane * 16 + a * 1024, xo, LD_AUX);
        __builtin_amdgcn_sched_barrier(0);
      }
      if (n > 0) {
#pragma unroll
        for (int a = 0; a < 4; ++a)
#pragma unroll
          for (int j = 0; j < 4; ++j)
            acc[a][j] = __builtin_amdgcn_mfma_i32_16x16x64_i8((i32x4)af[(n - 1) & 1][a], (i32x4)bf[(n - 1) & 1][j], acc[a][j], 0, 0, 0);
        __builtin_amdgcn_sched_barrier(0);
      }
    }
    STAMP(t * 12 + 8);
    int ln = lane;
    asm volatile("" : "+v"(ln));
#pragma unroll
    for (int a = 0; a < 4; ++a)
#pragma unroll
      for (int i = 0; i < 4; ++i)
#pragma unroll
        for (int jp = 0; jp < 2; ++jp) {
          const int lo = acc[a][2 * jp][i], hi = acc[a][2 * jp + 1][i];
          const u64 v = ((u64)(u32)(hi + (lo >> 31)) << 32) | (u32)lo;
          __hip_atomic_fetch_add(&sm.tile[t][a * 4 + i][jp][ln], v, __ATOMIC_RELAXED, __HIP_MEMORY_SCOPE_WORKGROUP);
        }
    int e0, e1;
    if (t + 1 == nphase) {
      __syncthreads();
      e0 = wave * 2; e1 = e0 + 2;
    } else {
      asm volatile("s_waitcnt lgkmcnt(0)" ::: "memory");
      u32 tk = 0;
      if (ln == 0) tk = __hip_atomic_fetch_add(&sm.arrive[t], 1u, __ATOMIC_RELAXED, __HIP_MEMORY_SCOPE_WORKGROUP);
      tk = __builtin_amdgcn_readfirstlane(tk);
      FENCE();
      e0 = 0; e1 = tk == 7 ? 16 : 0;
      if (t == 0 && tk == 7 && ln == 0) __hip_atomic_fetch_add(ctr, 1u, __ATOMIC_RELAXED, __HIP_MEMORY_SCOPE_AGENT);
    }
    STAMP(t * 12 + 9);
    if (e0 < e1) {
      const bool bad = LDS_LOAD(&sm.bad) != 0;
      const auto ro = __builtin_amdgcn_make_buffer_rsrc(P.out, 0, MS * GN * 4, 0x00020000);
      const int voff = (((ln >> 4) * 4) * GN + (ln & 15) * 4) * 4;
      for (int e = e0; e < e1; e += 2) {
#pragma unroll
        for (int h = 0; h < 2; ++h) {
          const u64 v0 = sm.tile[t][e + h][0][ln], v1 = sm.tile[t][e + h][1][ln];
          const int s0 = (int)(u32)v0, s2 = (int)(u32)v1;
          const int s1 = (int)(u32)(v0 >> 32) - (s0 >> 31), s3 = (int)(u32)(v1 >> 32) - (s2 >> 31);
          f32x4 y = f32x4{(float)s0 * scale, (float)s1 * scale, (float)s2 * scale, (float)s3 * scale};
          if (bad) y = f32x4{__builtin_nanf(""), __builtin_nanf(""), __builtin_nanf(""), __builtin_nanf("")};
          const int row = m * 64 + ((e + h) >> 2) * 16 + ((e + h) & 3);
          __builtin_amdgcn_raw_buffer_store_b128(__builtin_bit_cast(u32x4, y), ro, voff, (row * GN + grp * 64) * 4, OUT_AUX);
        }
      }
    }
    STAMP(t * 12 + 10);
  }
}

}

extern "C" void kernel_launch(int rank, int nranks, ncclComm_t comm,
                              hipStream_t stream,
                              void* const* d_in, const int* in_sizes, int n_in,
                              void* d_out, int out_size,
                              const int* in_dtypes, int out_dtype,
                              void* const* d_out_all,
                              void* const* d_ws_all, size_t ws_bytes) {
  if (nranks != NRANK || n_in != 4 || ws_bytes < WS_NEEDED || out_size != MS * GN ||
      in_sizes[0] != GM * KS || in_sizes[1] != KS * GN || (unsigned)rank >= (unsigned)NRANK) {
    fprintf(stderr, "kernel_launch: unexpected problem shape\n");
    return;
  }
  Params p;
  for (int i = 0; i < NRANK; ++i) p.ws[i] = (char*)d_ws_all[i];
  p.x = (const int8_t*)d_in[0];
  p.w = (const int8_t*)d_in[1];
  p.sx = (const float*)d_in[2];
  p.sw = (const float*)d_in[3];
  p.out = (float*)d_out;
  p.rank = rank;
  hipLaunchKernelGGL(gemm_gather_kernel, dim3(NWG), dim3(NTHREADS), 0, stream, p);
  HIP_CHECK(hipGetLastError());
}
```

```cpp
#include <hip/hip_runtime.h>
#include <rccl/rccl.h>
#include <cstdio>
#include <cstdint>

#define HIP_CHECK(expr) do { hipError_t _e = (expr); if (_e != hipSuccess) { \
    fprintf(stderr, "HIP error %s at %s:%d\n", hipGetErrorString(_e), __FILE__, __LINE__); \
    return; } } while (0)

namespace {

typedef unsigned int u32;
typedef unsigned long long u64;
typedef __attribute__((ext_vector_type(4))) int i32x4;
typedef __attribute__((ext_vector_type(4))) unsigned u32x4;
typedef __attribute__((ext_vector_type(4))) float f32x4;

#define PWAVES 4
#define PDEPTH 3
#define POLL_SLEEP 4
#define OUT_AUX 16
#define LD_AUX 16

constexpr int NRANK = 8;
constexpr int GM = 4096, GN = 4096, GK = 4096;
constexpr int MS = GM / NRANK;
constexpr int KS = GK / NRANK;
constexpr int NTHREADS = 512;
constexpr int NWG = 256;
constexpr int NXP = 56;
constexpr int NTILE = 144;
constexpr int PULL0 = NXP + NTILE;
constexpr int NSLOT = 8 * PWAVES;
constexpr int XROUND = 64 / NSLOT;
constexpr int NBLK = 576;
constexpr int NCH = 8;
constexpr int AUX_SYS = 17;
constexpr u32 TICKETS_PER_CALL = NBLK - 64 - 32 + 2 * 8;
static_assert(PWAVES == 2 || PWAVES == 4, "X push geometry");

constexpr size_t SRC_BYTES = (size_t)NBLK * 4096;
constexpr size_t PAR_BYTES = SRC_BYTES * NRANK;
constexpr size_t CTRL_OFF = 2 * PAR_BYTES;
constexpr size_t FLAG_OFF = CTRL_OFF + 4096;
constexpr size_t WS_NEEDED = FLAG_OFF + (size_t)NRANK * 72 * 128;
constexpr u64 DEADLINE_TICKS = 20000000ull;

struct Params {
  char* ws[NRANK];
  const int8_t* x;
  const int8_t* w;
  const float* sx;
  const float* sw;
  float* out;
  int rank;
};

#define STAMP(slot) do {} while (0)

#define WAIT_VM0() do { asm volatile("" ::: "memory"); __builtin_amdgcn_s_waitcnt(0x0F70); asm volatile("" ::: "memory"); } while (0)
#define WAIT_VMN(n) do { asm volatile("" ::: "memory"); __builtin_amdgcn_s_waitcnt(0x0F70 | (n)); asm volatile("" ::: "memory"); } while (0)
#define FENCE() asm volatile("" ::: "memory")
#define LDS_LOAD(p) __hip_atomic_load((p), __ATOMIC_RELAXED, __HIP_MEMORY_SCOPE_WORKGROUP)
#define LDS_STORE(p, v) __hip_atomic_store((p), (v), __ATOMIC_RELAXED, __HIP_MEMORY_SCOPE_WORKGROUP)

__device__ __forceinline__ void transpose4(u32 w0, u32 w1, u32 w2, u32 w3, u32& o0, u32& o1, u32& o2, u32& o3) {
  u32 t0 = __builtin_amdgcn_perm(w1, w0, 0x05010400u);
  u32 t1 = __builtin_amdgcn_perm(w1, w0, 0x07030602u);
  u32 t2 = __builtin_amdgcn_perm(w3, w2, 0x05010400u);
  u32 t3 = __builtin_amdgcn_perm(w3, w2, 0x07030602u);
  o0 = __builtin_amdgcn_perm(t2, t0, 0x05040100u);
  o1 = __builtin_amdgcn_perm(t2, t0, 0x07060302u);
  o2 = __builtin_amdgcn_perm(t3, t1, 0x05040100u);
  o3 = __builtin_amdgcn_perm(t3, t1, 0x07060302u);
}

__device__ __forceinline__ void decode_block(int n, int& row, int& c) {
  if (n < 64) { row = 64 + (n & 7); c = n >> 3; return; }
  const int i = n - 64;
  if (i < 384) { const int ph = i >= 192, j = i - ph * 192; c = j / 24; row = ph * 24 + j - c * 24; }
  else { const int j = i - 384; c = j >> 4; row = 48 + (j & 15); }
}
__device__ __forceinline__ void w_index(int grp, int& first, int& step) {
  if (grp < 24) { first = 64 + grp; step = 24; }
  else if (grp < 48) { first = 64 + 192 + grp - 24; step = 24; }
  else { first = 64 + 384 + grp - 48; step = 16; }
}

template <class R>
__device__ __forceinline__ void load_x_block(R rx, int dest, int n, int lane, u32x4 (&o)[4]) {
  const int soff = (dest * MS + (n & 7) * 64) * KS + (n >> 3) * 64;
#pragma unroll
  for (int a = 0; a < 4; ++a)
    o[a] = __builtin_amdgcn_raw_buffer_load_b128(rx, (lane & 15) * KS + (lane >> 4) * 16, soff + a * 16 * KS, 0);
}

template <class R>
__device__ __forceinline__ void load_w_block(R rw, int grp, int c, int lane, u32 (&raw)[16]) {
#pragma unroll
  for (int i = 0; i < 16; ++i)
    raw[i] = __builtin_amdgcn_raw_buffer_load_b32(rw, (lane >> 4) * 16 * GN + (lane & 15) * 4, (c * 64 + i) * GN + grp * 64, 0);
}
__device__ __forceinline__ void tile_w_block(const u32 (&raw)[16], u32x4 (&o)[4]) {
#pragma unroll
  for (int k = 0; k < 4; ++k) {
    u32 o0, o1, o2, o3;
    transpose4(raw[4 * k], raw[4 * k + 1], raw[4 * k + 2], raw[4 * k + 3], o0, o1, o2, o3);
    o[0][k] = o0; o[1][k] = o1; o[2][k] = o2; o[3][k] = o3;
  }
}

struct Smem {
  union {
    u64 tile[3][16][2][64];
    char fill[96 * 1024];
  };
  u32 arrive[4];
  u32 call;
  u32 bad;
  u32 pad[2];
};

__device__ __forceinline__ bool deadline_hit(u32 spins, u64& t0, u32* tmo, u32 call) {
  if (spins == 0) t0 = __builtin_amdgcn_s_memrealtime();
  if ((spins & 63) != 63) return false;
  if (__hip_atomic_load(tmo, __ATOMIC_RELAXED, __HIP_MEMORY_SCOPE_AGENT) == call ||
      __builtin_amdgcn_s_memrealtime() - t0 > DEADLINE_TICKS) {
    __hip_atomic_store(tmo, call, __ATOMIC_RELAXED, __HIP_MEMORY_SCOPE_AGENT);
    return true;
  }
  return false;
}

__device__ __forceinline__ u32 poll_ready(u32* fptr, int nlanes, u32 todo, int lane, u32* tmo, u32 call) {
  u64 t0 = 0;
  for (u32 spins = 0;; ++spins) {
    u32 f = call;
    if (lane < nlanes) f = __hip_atomic_load(fptr, __ATOMIC_RELAXED, __HIP_MEMORY_SCOPE_SYSTEM);
    const u64 b = __ballot((int)(f - call) >= 0);
    const u32 ready = (u32)(b & (b >> 8)) & todo;
    if (ready) { FENCE(); return ready; }
    if (deadline_hit(spins, t0, tmo, call)) return 0;
    __builtin_amdgcn_s_sleep(POLL_SLEEP);
  }
}

__global__ void __launch_bounds__(NTHREADS) gemm_gather_kernel(Params P) {
  __shared__ __attribute__((aligned(16))) Smem sm;
  const int tid = threadIdx.x;
  const int lane = tid & 63;
  const int wave = __builtin_amdgcn_readfirstlane(tid >> 6);
  const int bid = blockIdx.x;
  const int rank = P.rank;
  char* myws = P.ws[rank];
  u32* ctr = (u32*)(myws + CTRL_OFF);
  u32* tmo = (u32*)(myws + CTRL_OFF + 256);
  u32* tiled = (u32*)(myws + CTRL_OFF + 512);
  const auto rx = __builtin_amdgcn_make_buffer_rsrc((void*)P.x, 0, GM * KS, 0x00020000);
  const auto rl = __builtin_amdgcn_make_buffer_rsrc(myws, 0, (int)WS_NEEDED, 0x00020000);
  const auto rw = __builtin_amdgcn_make_buffer_rsrc((void*)P.w, 0, KS * GN, 0x00020000);

  const int lane0 = lane == 0 ? 0 : 0x7FFFFFF0;
  if (bid >= PULL0) {
    const u32 cnt = __hip_atomic_load(ctr, __ATOMIC_RELAXED, __HIP_MEMORY_SCOPE_AGENT);
    const u32 call = (u32)__builtin_amdgcn_readfirstlane(cnt) / (u32)NWG + 1;
    if (wave == 0) {
      const int peer = (rank + 1 + ((bid - PULL0) >> 3)) & 7;
      const auto rs = __builtin_amdgcn_make_buffer_rsrc(P.ws[peer], 0, (int)WS_NEEDED, 0x00020000);
      const u32 dbase = (u32)((call & 1) ? PAR_BYTES : 0) + (u32)peer * (u32)SRC_BYTES;
      const u32 fbase = (u32)FLAG_OFF + (u32)peer * (72 * 128);
      bool ok = true;
      {
        u32* rdy = (u32*)(myws + CTRL_OFF + 2048) + peer * 32;
        u64 t0 = 0;
        for (u32 spins = 0; (int)(__hip_atomic_load(rdy, __ATOMIC_RELAXED, __HIP_MEMORY_SCOPE_SYSTEM) - call) < 0; ++spins) {
          if (deadline_hit(spins, t0, tmo, call)) { ok = false; break; }
          __builtin_amdgcn_s_sleep(2);
        }
        FENCE();
      }
      STAMP(3);
      constexpr int OOB = 0x7FFF0000;
      auto fetch = [&](u32 n, u32x4 (&d)[4]) __attribute__((always_inline)) {
        const bool v = n < (u32)NBLK;
        const int vo = v ? lane * 16 : OOB;
        const u32 so = v ? dbase + n * 4096 : 0;
#pragma unroll
        for (int j = 0; j < 4; ++j) d[j] = __builtin_amdgcn_raw_buffer_load_b128(rs, vo + j * 1024, so, AUX_SYS);
        FENCE();
      };
      auto put = [&](u32 n, const u32x4 (&d)[4]) __attribute__((always_inline)) {
        const bool v = n < (u32)NBLK;
        const int vo = v ? lane * 16 : OOB;
        const u32 so = v ? dbase + n * 4096 : 0;
#pragma unroll
        for (int j = 0; j < 4; ++j) __builtin_amdgcn_raw_buffer_store_b128(d[j], rl, vo + j * 1024, so, AUX_SYS);
        FENCE();
      };
      auto flag = [&](u32 n) __attribute__((always_inline)) {
        const bool v = n < (u32)NBLK;
        const u32 i = v ? n - 64 : 0;
        const u32 ph = i >= 384 ? 2 : (i >= 192 ? 1 : 0), j = i - ph * 192;
        const u32 c = ph == 2 ? j >> 4 : (j * 2731) >> 16;
        const u32 row = ph * 24 + j - c * (ph == 2 ? 16 : 24);
        const u32 foff = __builtin_amdgcn_readfirstlane(fbase + row * 128 + c * 4);
        __builtin_amdgcn_raw_buffer_store_b32(call, rl, v ? lane0 : OOB, foff, AUX_SYS);
        FENCE();
      };
      if (ok) {
        const int qoff = (int)CTRL_OFF + 3072 + ((bid - PULL0) >> 3) * 128;
        const u32 qbase = (call - 1) * TICKETS_PER_CALL - (64 + 32);
        auto ticket = [&](bool v) __attribute__((always_inline)) -> u32 {
          const u32 tk = (u32)__builtin_amdgcn_raw_ptr_buffer_atomic_add_i32(1, rl, v ? lane0 : OOB, qoff, 0);
          FENCE();
          return tk;
        };
        u32x4 d0[4], d1[4];
        u32 i0 = 64 + (bid & 7), i1 = i0 + 8, i2 = i0 + 16, i3 = i0 + 24;
        fetch(i0, d0);
        fetch(i1, d1);
        WAIT_VMN(4);  put(i0, d0); u32 tk0 = ticket(true); fetch(i2, d0); flag(NBLK);
        WAIT_VMN(10); put(i1, d1); u32 tk1 = ticket(true); fetch(i3, d1); flag(NBLK);
        auto step = [&](u32x4 (&d)[4], u32& tk) __attribute__((always_inline)) {
          WAIT_VMN(11);
          const u32 i4 = i2 < (u32)NBLK ? (u32)__builtin_amdgcn_readfirstlane(tk) - qbase : (u32)NBLK;
          put(i2, d);
          tk = ticket(i4 < (u32)NBLK);
          fetch(i4, d);
          flag(i0);
          i0 = i1; i1 = i2; i2 = i3; i3 = i4;
        };
#pragma unroll 1
        do {
          step(d0, tk0);
          step(d1, tk1);
        } while (i0 < (u32)NBLK);
      } else {
        u32* queue = (u32*)(myws + CTRL_OFF + 3072 + ((bid - PULL0) >> 3) * 128);
        const u32 qbase = (call - 1) * TICKETS_PER_CALL - (64 + 32);
        if (lane == 0)
          for (int inv = 0; inv < 2;)
            inv += __hip_atomic_fetch_add(queue, 1u, __ATOMIC_RELAXED, __HIP_MEMORY_SCOPE_AGENT) - qbase >= (u32)NBLK;
      }
      STAMP(2);
    } else if (wave == 1 && bid == PULL0) {
      const u32 target = call * (NTILE / 8);
      u64 t0 = 0;
      bool ok = true;
      for (u32 spins = 0;; ++spins) {
        u32 f = target;
        if (lane < 8) f = __hip_atomic_load(tiled + lane * 32, __ATOMIC_RELAXED, __HIP_MEMORY_SCOPE_AGENT);
        if (__all((int)(f - target) >= 0)) break;
        if (deadline_hit(spins, t0, tmo, call)) { ok = false; break; }
        __builtin_amdgcn_s_sleep(1);
      }
      FENCE();
      if (ok) {
#pragma unroll
        for (int i = 1; i < NRANK; ++i) {
          const auto rs = __builtin_amdgcn_make_buffer_rsrc(P.ws[(rank + i) & 7], 0, (int)WS_NEEDED, 0x00020000);
          __builtin_amdgcn_raw_buffer_store_b32(call, rs, lane0, (int)CTRL_OFF + 2048 + rank * 128, AUX_SYS);
        }
      }
      STAMP(36);
    }
    __syncthreads();
    if (tid == 0) __hip_atomic_fetch_add(ctr, 1u, __ATOMIC_RELAXED, __HIP_MEMORY_SCOPE_AGENT);
    return;
  }

  const int q = bid < NXP ? NTILE + bid - 8 : bid - NXP;
  const int xcd = bid & 7;
  const int m = (q >> 3) & 7;
  const int gl = q >> 6;
  const int src = (rank - wave) & 7;
  auto init_lds = [&]() __attribute__((always_inline)) {
    u64* z = &sm.tile[0][0][0][0];
#pragma unroll
    for (int i = 0; i < 3 * 16 * 2 * 64 / NTHREADS; ++i) z[i * NTHREADS + tid] = 0;
    if (tid < 4) sm.arrive[tid] = 0;
    if (tid == 4) sm.bad = 0;
    asm volatile("s_waitcnt lgkmcnt(0)" ::: "memory");
    __builtin_amdgcn_s_barrier();
  };
  u32 call;
  if (bid < NXP && wave < PWAVES) {
    const int peer = (rank + 1 + (bid >> 3)) & 7;
    const int slot = (bid & 7) * PWAVES + wave;
    u32x4 buf[XROUND][4];
    load_x_block(rx, peer, slot, lane, buf[0]);
    const u32 cnt = __hip_atomic_load(ctr, __ATOMIC_RELAXED, __HIP_MEMORY_SCOPE_AGENT);
#pragma unroll
    for (int r = 1; r < XROUND; ++r) load_x_block(rx, peer, r * NSLOT + slot, lane, buf[r]);
    FENCE();
    call = (u32)__builtin_amdgcn_readfirstlane(cnt) / (u32)NWG + 1;
    const auto rs = __builtin_amdgcn_make_buffer_rsrc(P.ws[peer], 0, (int)WS_NEEDED, 0x00020000);
    const u32 dbase = (u32)((call & 1) ? PAR_BYTES : 0) + (u32)rank * (u32)SRC_BYTES;
    const u32 fbase = (u32)FLAG_OFF + (u32)rank * (72 * 128);
#pragma unroll
    for (int r = 0; r <= XROUND; ++r) {
      if (r > 0) {
        const int n = (r - 1) * NSLOT + slot;
        WAIT_VM0();
        __builtin_amdgcn_raw_buffer_store_b32(call, rs, lane0, fbase + (u32)(64 + (n & 7)) * 128 + (u32)(n >> 3) * 4, AUX_SYS);
      }
      if (r < XROUND) {
#pragma unroll
        for (int j = 0; j < 4; ++j)
          __builtin_amdgcn_raw_buffer_store_b128(buf[r][j], rs, lane * 16 + j * 1024, dbase + (u32)(r * NSLOT + slot) * 4096, AUX_SYS);
      }
      FENCE();
      if (r == 0) {
        init_lds();
      }
    }
    STAMP(39);
  } else if (bid >= NXP && wave >= 4) {
    const int n = (64 + (wave - 4) * NTILE + q) % NBLK;
    int row, c;
    decode_block(n, row, c);
    u32x4 o[4];
    u32 raw[16];
    if (n < 64) load_x_block(rx, rank, n, lane, o);
    else load_w_block(rw, row, c, lane, raw);
    const u32 cnt = __hip_atomic_load(ctr, __ATOMIC_RELAXED, __HIP_MEMORY_SCOPE_AGENT);
    FENCE();
    init_lds();
    call = (u32)__builtin_amdgcn_readfirstlane(cnt) / (u32)NWG + 1;
    if (n >= 64) tile_w_block(raw, o);
    const u32 dbase = (u32)((call & 1) ? PAR_BYTES : 0) + (u32)rank * (u32)SRC_BYTES;
#pragma unroll
    for (int j = 0; j < 4; ++j)
      __builtin_amdgcn_raw_buffer_store_b128(o[j], rl, lane * 16 + j * 1024, dbase + (u32)n * 4096, AUX_SYS);
    WAIT_VM0();
    if (lane == 0) {
      if (__hip_atomic_fetch_add(&sm.arrive[3], 1u, __ATOMIC_RELAXED, __HIP_MEMORY_SCOPE_WORKGROUP) == 3)
        __hip_atomic_fetch_add(tiled + xcd * 32, 1u, __ATOMIC_RELAXED, __HIP_MEMORY_SCOPE_AGENT);
      __builtin_amdgcn_raw_buffer_store_b32(call, rl, 0, (u32)FLAG_OFF + (u32)rank * (72 * 128) + (u32)row * 128 + (u32)c * 4, AUX_SYS);
    }
    STAMP(36);
  } else {
    const u32 cnt = __hip_atomic_load(ctr, __ATOMIC_RELAXED, __HIP_MEMORY_SCOPE_AGENT);
    FENCE();
    init_lds();
    call = (u32)__builtin_amdgcn_readfirstlane(cnt) / (u32)NWG + 1;
  }
  if (bid < 8) {
    __syncthreads();
    if (tid == 0) __hip_atomic_fetch_add(ctr, 1u, __ATOMIC_RELAXED, __HIP_MEMORY_SCOPE_AGENT);
    return;
  }
  const float scale = P.sx[0] * P.sw[0];
  const u32 sbase = (u32)((call & 1) ? PAR_BYTES : 0) + (u32)src * (u32)SRC_BYTES;
  u32* fsrc = (u32*)(myws + FLAG_OFF + (size_t)src * (72 * 128));
  const int nphase = gl < 2 ? 3 : 2;

#pragma unroll 1
  for (int t = 0; t < nphase; ++t) {
    const int grp = t < 2 ? t * 24 + xcd * 3 + gl : 48 + xcd * 2 + gl;
    i32x4 acc[4][4];
#pragma unroll
    for (int a = 0; a < 4; ++a)
#pragma unroll
      for (int b = 0; b < 4; ++b) acc[a][b] = i32x4{0, 0, 0, 0};
    int wfirst, wstep;
    w_index(grp, wfirst, wstep);
    u32* fptr = fsrc + (lane < 8 ? grp : 64 + m) * 32 + (lane & 7);
    u32 todo = 0xFF, ready = 0;
    u32x4 af[2][4], bf[2][4];
#pragma unroll
    for (int n = 0; n <= NCH; ++n) {
      if (n < NCH) {
        if (!ready) {
          ready = poll_ready(fptr, t == 0 ? 16 : 8, todo, lane, tmo, call);
          if (!ready) {
            LDS_STORE(&sm.bad, 1u);
            ready = todo;
          }
        }
        const int c = __builtin_ctz(ready);
        ready &= ready - 1;
        todo &= ~(1u << c);
        STAMP(t * 12 + n);
        const u32 xo = sbase + (u32)(c * 8 + m) * 4096;
        const u32 wo = __builtin_amdgcn_readfirstlane(sbase + (u32)(wfirst + c * wstep) * 4096);
#pragma unroll
        for (int j = 0; j < 4; ++j) bf[n & 1][j] = __builtin_amdgcn_raw_buffer_load_b128(rl, lane * 16 + j * 1024, wo, LD_AUX);
#pragma unroll
        for (int a = 0; a < 4; ++a) af[n & 1][a] = __builtin_amdgcn_raw_buffer_load_b128(rl, lane * 16 + a * 1024, xo, LD_AUX);
        __builtin_amdgcn_sched_barrier(0);
      }
      if (n > 0) {
#pragma unroll
        for (int a = 0; a < 4; ++a)
#pragma unroll
          for (int j = 0; j < 4; ++j)
            acc[a][j] = __builtin_amdgcn_mfma_i32_16x16x64_i8((i32x4)af[(n - 1) & 1][a], (i32x4)bf[(n - 1) & 1][j], acc[a][j], 0, 0, 0);
        __builtin_amdgcn_sched_barrier(0);
      }
    }
    STAMP(t * 12 + 8);
    int ln = lane;
    asm volatile("" : "+v"(ln));
#pragma unroll
    for (int a = 0; a < 4; ++a)
#pragma unroll
      for (int i = 0; i < 4; ++i)
#pragma unroll
        for (int jp = 0; jp < 2; ++jp) {
          const int lo = acc[a][2 * jp][i], hi = acc[a][2 * jp + 1][i];
          const u64 v = ((u64)(u32)(hi + (lo >> 31)) << 32) | (u32)lo;
          __hip_atomic_fetch_add(&sm.tile[t][a * 4 + i][jp][ln], v, __ATOMIC_RELAXED, __HIP_MEMORY_SCOPE_WORKGROUP);
        }
    int e0, e1;
    if (t + 1 == nphase) {
      __syncthreads();
      e0 = wave * 2; e1 = e0 + 2;
    } else {
      asm volatile("s_waitcnt lgkmcnt(0)" ::: "memory");
      u32 tk = 0;
      if (ln == 0) tk = __hip_atomic_fetch_add(&sm.arrive[t], 1u, __ATOMIC_RELAXED, __HIP_MEMORY_SCOPE_WORKGROUP);
      tk = __builtin_amdgcn_readfirstlane(tk);
      FENCE();
      e0 = 0; e1 = tk == 7 ? 16 : 0;
      if (t == 0 && tk == 7 && ln == 0) __hip_atomic_fetch_add(ctr, 1u, __ATOMIC_RELAXED, __HIP_MEMORY_SCOPE_AGENT);
    }
    STAMP(t * 12 + 9);
    if (e0 < e1) {
      const bool bad = LDS_LOAD(&sm.bad) != 0;
      const auto ro = __builtin_amdgcn_make_buffer_rsrc(P.out, 0, MS * GN * 4, 0x00020000);
      const int voff = (((ln >> 4) * 4) * GN + (ln & 15) * 4) * 4;
      for (int e = e0; e < e1; e += 2) {
#pragma unroll
        for (int h = 0; h < 2; ++h) {
          const u64 v0 = sm.tile[t][e + h][0][ln], v1 = sm.tile[t][e + h][1][ln];
          const int s0 = (int)(u32)v0, s2 = (int)(u32)v1;
          const int s1 = (int)(u32)(v0 >> 32) - (s0 >> 31), s3 = (int)(u32)(v1 >> 32) - (s2 >> 31);
          f32x4 y = f32x4{(float)s0 * scale, (float)s1 * scale, (float)s2 * scale, (float)s3 * scale};
          if (bad) y = f32x4{__builtin_nanf(""), __builtin_nanf(""), __builtin_nanf(""), __builtin_nanf("")};
          const int row = m * 64 + ((e + h) >> 2) * 16 + ((e + h) & 3);
          __builtin_amdgcn_raw_buffer_store_b128(__builtin_bit_cast(u32x4, y), ro, voff, (row * GN + grp * 64) * 4, OUT_AUX);
        }
      }
    }
    STAMP(t * 12 + 10);
  }
}

}

extern "C" void kernel_launch(int rank, int nranks, ncclComm_t comm,
                              hipStream_t stream,
                              void* const* d_in, const int* in_sizes, int n_in,
                              void* d_out, int out_size,
                              const int* in_dtypes, int out_dtype,
                              void* const* d_out_all,
                              void* const* d_ws_all, size_t ws_bytes) {
  if (nranks != NRANK || n_in != 4 || ws_bytes < WS_NEEDED || out_size != MS * GN ||
      in_sizes[0] != GM * KS || in_sizes[1] != KS * GN || (unsigned)rank >= (unsigned)NRANK) {
    fprintf(stderr, "kernel_launch: unexpected problem shape\n");
    return;
  }
  Params p;
  for (int i = 0; i < NRANK; ++i) p.ws[i] = (char*)d_ws_all[i];
  p.x = (const int8_t*)d_in[0];
  p.w = (const int8_t*)d_in[1];
  p.sx = (const float*)d_in[2];
  p.sw = (const float*)d_in[3];
  p.out = (float*)d_out;
  p.rank = rank;
  hipLaunchKernelGGL(gemm_gather_kernel, dim3(NWG), dim3(NTHREADS), 0, stream, p);
  HIP_CHECK(hipGetLastError());
}
```

```cpp
#include <hip/hip_runtime.h>
#include <rccl/rccl.h>
#include <cstdio>
#include <cstdint>

#define HIP_CHECK(expr) do { hipError_t _e = (expr); if (_e != hipSuccess) { \
    fprintf(stderr, "HIP error %s at %s:%d\n", hipGetErrorString(_e), __FILE__, __LINE__); \
    return; } } while (0)

namespace {

typedef unsigned int u32;
typedef unsigned long long u64;
typedef __attribute__((ext_vector_type(4))) int i32x4;
typedef __attribute__((ext_vector_type(4))) unsigned u32x4;
typedef __attribute__((ext_vector_type(4))) float f32x4;

#define PWAVES 4
#define PDEPTH 3
#define PGATE 500
#define EARLY_EXIT 1
#define POLL_SLEEP 4
#define OUT_AUX 16
#define LD_AUX 17

constexpr int NRANK = 8;
constexpr int GM = 4096, GN = 4096, GK = 4096;
constexpr int MS = GM / NRANK;
constexpr int KS = GK / NRANK;
constexpr int NTHREADS = 512;
constexpr int NWG = 256;
constexpr int NXP = 56;
constexpr int NTILE = 144;
constexpr int PULL0 = NXP + NTILE;
constexpr int NSLOT = 8 * PWAVES;
constexpr int XROUND = 64 / NSLOT;
constexpr int NBLK = 576;
constexpr int NCH = 8;
constexpr int AUX_SYS = 17;
constexpr int OOB = 0x7FFF0000;
constexpr u32 TICKETS_PER_CALL = NBLK - 64 - 32 + 2 * 8;
static_assert(PWAVES == 2 || PWAVES == 4, "X push geometry");

constexpr size_t SRC_BYTES = (size_t)NBLK * 4096;
constexpr size_t PAR_BYTES = SRC_BYTES * NRANK;
constexpr size_t CTRL_OFF = 2 * PAR_BYTES;
constexpr size_t FLAG_OFF = CTRL_OFF + 4096;
constexpr size_t WS_NEEDED = FLAG_OFF + (size_t)NRANK * 72 * 128;
constexpr u64 DEADLINE_TICKS = 20000000ull;

struct Params {
  char* ws[NRANK];
  const int8_t* x;
  const int8_t* w;
  const float* sx;
  const float* sw;
  float* out;
  int rank;
};

#define STAMP(slot) do {} while (0)

#define WAIT_VM0() do { asm volatile("" ::: "memory"); __builtin_amdgcn_s_waitcnt(0x0F70); asm volatile("" ::: "memory"); } while (0)
#define WAIT_VMN(n) do { asm volatile("" ::: "memory"); __builtin_amdgcn_s_waitcnt(0x0F70 | (n)); asm volatile("" ::: "memory"); } while (0)
#define FENCE() asm volatile("" ::: "memory")
#define LDS_LOAD(p) __hip_atomic_load((p), __ATOMIC_RELAXED, __HIP_MEMORY_SCOPE_WORKGROUP)
#define LDS_STORE(p, v) __hip_atomic_store((p), (v), __ATOMIC_RELAXED, __HIP_MEMORY_SCOPE_WORKGROUP)

template <int AUX, class R>
__device__ __forceinline__ void store16(u32x4 d, R r, int voff, u32 soff) {
  __builtin_amdgcn_raw_buffer_store_b128(d, r, voff, soff, AUX);
  asm volatile("s_nop 1" ::"v"(d) : "memory");
}
template <int AUX, class R>
__device__ __forceinline__ void store_block(const u32x4 (&d)[4], R r, int voff, u32 soff) {
#pragma unroll
  for (int j = 0; j < 4; ++j) __builtin_amdgcn_raw_buffer_store_b128(d[j], r, voff + j * 1024, soff, AUX);
  asm volatile("s_nop 1" ::"v"(d[0]), "v"(d[1]), "v"(d[2]), "v"(d[3]) : "memory");
}

__device__ __forceinline__ void transpose4(u32 w0, u32 w1, u32 w2, u32 w3, u32& o0, u32& o1, u32& o2, u32& o3) {
  u32 t0 = __builtin_amdgcn_perm(w1, w0, 0x05010400u);
  u32 t1 = __builtin_amdgcn_perm(w1, w0, 0x07030602u);
  u32 t2 = __builtin_amdgcn_perm(w3, w2, 0x05010400u);
  u32 t3 = __builtin_amdgcn_perm(w3, w2, 0x07030602u);
  o0 = __builtin_amdgcn_perm(t2, t0, 0x05040100u);
  o1 = __builtin_amdgcn_perm(t2, t0, 0x07060302u);
  o2 = __builtin_amdgcn_perm(t3, t1, 0x05040100u);
  o3 = __builtin_amdgcn_perm(t3, t1, 0x07060302u);
}

__device__ __forceinline__ void decode_block(int n, int& row, int& c) {
  if (n < 64) { row = 64 + (n & 7); c = n >> 3; return; }
  const int i = n - 64;
  if (i < 384) { const int ph = i >= 192, j = i - ph * 192; c = j / 24; row = ph * 24 + j - c * 24; }
  else { const int j = i - 384; c = j >> 4; row = 48 + (j & 15); }
}
__device__ __forceinline__ void w_index(int grp, int& first, int& step) {
  if (grp < 24) { first = 64 + grp; step = 24; }
  else if (grp < 48) { first = 64 + 192 + grp - 24; step = 24; }
  else { first = 64 + 384 + grp - 48; step = 16; }
}

template <class R>
__device__ __forceinline__ void load_x_block(R rx, int dest, int n, int lane, u32x4 (&o)[4]) {
  const int soff = (dest * MS + (n & 7) * 64) * KS + (n >> 3) * 64;
#pragma unroll
  for (int a = 0; a < 4; ++a)
    o[a] = __builtin_amdgcn_raw_buffer_load_b128(rx, (lane & 15) * KS + (lane >> 4) * 16, soff + a * 16 * KS, 0);
}

template <class R>
__device__ __forceinline__ void load_w_block(R rw, int grp, int c, int lane, u32 (&raw)[16]) {
#pragma unroll
  for (int i = 0; i < 16; ++i)
    raw[i] = __builtin_amdgcn_raw_buffer_load_b32(rw, (lane >> 4) * 16 * GN + (lane & 15) * 4, (c * 64 + i) * GN + grp * 64, 0);
}
__device__ __forceinline__ void tile_w_block(const u32 (&raw)[16], u32x4 (&o)[4]) {
#pragma unroll
  for (int k = 0; k < 4; ++k) {
    u32 o0, o1, o2, o3;
    transpose4(raw[4 * k], raw[4 * k + 1], raw[4 * k + 2], raw[4 * k + 3], o0, o1, o2, o3);
    o[0][k] = o0; o[1][k] = o1; o[2][k] = o2; o[3][k] = o3;
  }
}

struct Smem {
  union {
    u64 tile[3][16][2][64];
    char fill[96 * 1024];
  };
  u32 arrive[4];
  u32 call;
  u32 bad;
  u32 pad[2];
};

__device__ __forceinline__ bool deadline_hit(u32 spins, u64& t0, u32* tmo, u32 call) {
  if (spins == 0) t0 = __builtin_amdgcn_s_memrealtime();
  if ((spins & 63) != 63) return false;
  if (__hip_atomic_load(tmo, __ATOMIC_RELAXED, __HIP_MEMORY_SCOPE_AGENT) == call ||
      __builtin_amdgcn_s_memrealtime() - t0 > DEADLINE_TICKS) {
    __hip_atomic_store(tmo, call, __ATOMIC_RELAXED, __HIP_MEMORY_SCOPE_AGENT);
    return true;
  }
  return false;
}

__device__ __forceinline__ u32 poll_ready(u32* fptr, int nlanes, u32 todo, int lane, u32* tmo, u32 call) {
  u64 t0 = 0;
  for (u32 spins = 0;; ++spins) {
    u32 f = call;
    if (lane < nlanes) f = __hip_atomic_load(fptr, __ATOMIC_RELAXED, __HIP_MEMORY_SCOPE_SYSTEM);
    const u64 b = __ballot((int)(f - call) >= 0);
    const u32 ready = (u32)(b & (b >> 8)) & todo;
    if (ready) { FENCE(); return ready; }
    if (deadline_hit(spins, t0, tmo, call)) return 0;
    __builtin_amdgcn_s_sleep(POLL_SLEEP);
  }
}

__global__ void __launch_bounds__(NTHREADS) gemm_gather_kernel(Params P) {
  __shared__ __attribute__((aligned(16))) Smem sm;
  const int tid = threadIdx.x;
  const int lane = tid & 63;
  const int wave = __builtin_amdgcn_readfirstlane(tid >> 6);
  const int bid = blockIdx.x;
  const int rank = P.rank;
  char* myws = P.ws[rank];
  u32* ctr = (u32*)(myws + CTRL_OFF);
  u32* tmo = (u32*)(myws + CTRL_OFF + 256);
  u32* tiled = (u32*)(myws + CTRL_OFF + 512);
  const auto rx = __builtin_amdgcn_make_buffer_rsrc((void*)P.x, 0, GM * KS, 0x00020000);
  const auto rl = __builtin_amdgcn_make_buffer_rsrc(myws, 0, (int)WS_NEEDED, 0x00020000);
  const auto rw = __builtin_amdgcn_make_buffer_rsrc((void*)P.w, 0, KS * GN, 0x00020000);

  const int lane0 = lane == 0 ? 0 : 0x7FFFFFF0;
  if (bid >= PULL0) {
    if (wave >= 2 || (wave == 1 && bid != PULL0)) return;
    const u64 tstart = __builtin_amdgcn_s_memrealtime();
    const u32 cnt = __hip_atomic_load(ctr, __ATOMIC_RELAXED, __HIP_MEMORY_SCOPE_AGENT);
    const u32 call = (u32)__builtin_amdgcn_readfirstlane(cnt) / (u32)NWG + 1;
    if (wave == 0) {
      while (__builtin_amdgcn_s_memrealtime() - tstart < PGATE) __builtin_amdgcn_s_sleep(4);
      const int peer = (rank + 1 + ((bid - PULL0) >> 3)) & 7;
      const auto rs = __builtin_amdgcn_make_buffer_rsrc(P.ws[peer], 0, (int)WS_NEEDED, 0x00020000);
      const u32 dbase = (u32)((call & 1) ? PAR_BYTES : 0) + (u32)peer * (u32)SRC_BYTES;
      const u32 fbase = (u32)FLAG_OFF + (u32)peer * (72 * 128);
      bool ok = true;
      {
        u32* rdy = (u32*)(myws + CTRL_OFF + 2048) + peer * 32;
        u64 t0 = 0;
        for (u32 spins = 0; (int)(__hip_atomic_load(rdy, __ATOMIC_RELAXED, __HIP_MEMORY_SCOPE_SYSTEM) - call) < 0; ++spins) {
          if (deadline_hit(spins, t0, tmo, call)) { ok = false; break; }
          __builtin_amdgcn_s_sleep(2);
        }
        FENCE();
      }
      STAMP(3);
      auto fetch = [&](u32 n, u32x4 (&d)[4]) __attribute__((always_inline)) {
        const bool v = n < (u32)NBLK;
        const int vo = v ? lane * 16 : OOB;
        const u32 so = v ? dbase + n * 4096 : 0;
#pragma unroll
        for (int j = 0; j < 4; ++j) d[j] = __builtin_amdgcn_raw_buffer_load_b128(rs, vo + j * 1024, so, AUX_SYS);
        FENCE();
      };
      auto put = [&](u32 n, const u32x4 (&d)[4]) __attribute__((always_inline)) {
        const bool v = n < (u32)NBLK;
        const int vo = v ? lane * 16 : OOB;
        const u32 so = v ? dbase + n * 4096 : 0;
        store_block<AUX_SYS>(d, rl, vo, so);
      };
      auto flag = [&](u32 n) __attribute__((always_inline)) {
        const bool v = n < (u32)NBLK;
        const u32 i = v ? n - 64 : 0;
        const u32 ph = i >= 384 ? 2 : (i >= 192 ? 1 : 0), j = i - ph * 192;
        const u32 c = ph == 2 ? j >> 4 : (j * 2731) >> 16;
        const u32 row = ph * 24 + j - c * (ph == 2 ? 16 : 24);
        const u32 foff = __builtin_amdgcn_readfirstlane(fbase + row * 128 + c * 4);
        __builtin_amdgcn_raw_buffer_store_b32(call, rl, v ? lane0 : OOB, foff, AUX_SYS);
        FENCE();
      };
      if (ok) {
        const int qoff = (int)CTRL_OFF + 3072 + ((bid - PULL0) >> 3) * 128;
        const u32 qbase = (call - 1) * TICKETS_PER_CALL - (64 + 32);
        auto ticket = [&](bool v) __attribute__((always_inline)) -> u32 {
          const u32 tk = (u32)__builtin_amdgcn_raw_ptr_buffer_atomic_add_i32(1, rl, v ? lane0 : OOB, qoff, 0);
          FENCE();
          return tk;
        };
        u32x4 d0[4], d1[4];
        u32 i0 = 64 + (bid & 7), i1 = i0 + 8, i2 = i0 + 16, i3 = i0 + 24;
        fetch(i0, d0);
        fetch(i1, d1);
        WAIT_VMN(4); put(i0, d0); u32 tk0 = ticket(true); fetch(i2, d0);
        WAIT_VMN(9); put(i1, d1); u32 tk1 = ticket(true); fetch(i3, d1);
#define PULL_STEP(D, TK, CNT) {                                                                          \
          const bool drain = i3 >= (u32)NBLK;                                                            \
          if (!drain) WAIT_VMN(CNT); else WAIT_VM0();                                                    \
            \
          const u32 i4 = i2 < (u32)NBLK ? (u32)__builtin_amdgcn_readfirstlane(TK) - qbase : (u32)NBLK;   \
          put(i2, D);                                                                                    \
          TK = ticket(i4 < (u32)NBLK);                                                                   \
          fetch(i4, D);                                                                                  \
          flag(i0);                                                                                      \
          if (drain) { flag(i1); i1 = NBLK; }                                                            \
          i0 = i1; i1 = i2; i2 = i3; i3 = i4;                                                            \
        }
        PULL_STEP(d0, tk0, 9)
        PULL_STEP(d1, tk1, 10)
#pragma unroll 1
        do {
          PULL_STEP(d0, tk0, 11)
          PULL_STEP(d1, tk1, 11)
        } while (i0 < (u32)NBLK || i1 < (u32)NBLK);
#undef PULL_STEP
      } else {
        u32* queue = (u32*)(myws + CTRL_OFF + 3072 + ((bid - PULL0) >> 3) * 128);
        const u32 qbase = (call - 1) * TICKETS_PER_CALL - (64 + 32);
        if (lane == 0)
          for (int inv = 0; inv < 2;)
            inv += __hip_atomic_fetch_add(queue, 1u, __ATOMIC_RELAXED, __HIP_MEMORY_SCOPE_AGENT) - qbase >= (u32)NBLK;
      }
      STAMP(2);
    } else if (wave == 1 && bid == PULL0) {
      const u32 target = call * (NTILE / 8);
      u64 t0 = 0;
      bool ok = true;
      for (u32 spins = 0;; ++spins) {
        u32 f = target;
        if (lane < 8) f = __hip_atomic_load(tiled + lane * 32, __ATOMIC_RELAXED, __HIP_MEMORY_SCOPE_AGENT);
        if (__all((int)(f - target) >= 0)) break;
        if (deadline_hit(spins, t0, tmo, call)) { ok = false; break; }
        __builtin_amdgcn_s_sleep(1);
      }
      FENCE();
      if (ok) {
#pragma unroll
        for (int i = 1; i < NRANK; ++i) {
          const auto rs = __builtin_amdgcn_make_buffer_rsrc(P.ws[(rank + i) & 7], 0, (int)WS_NEEDED, 0x00020000);
          __builtin_amdgcn_raw_buffer_store_b32(call, rs, lane0, (int)CTRL_OFF + 2048 + rank * 128, AUX_SYS);
        }
      }
      STAMP(36);
    }
    __syncthreads();
    if (tid == 0) __hip_atomic_fetch_add(ctr, 1u, __ATOMIC_RELAXED, __HIP_MEMORY_SCOPE_AGENT);
    return;
  }

  const int q = bid < NXP ? NTILE + bid - 8 : bid - NXP;
  const int xcd = bid & 7;
  const int m = (q >> 3) & 7;
  const int gl = q >> 6;
  const int src = (rank - wave) & 7;
  auto init_lds = [&]() __attribute__((always_inline)) {
    u64* z = &sm.tile[0][0][0][0];
#pragma unroll
    for (int i = 0; i < 3 * 16 * 2 * 64 / NTHREADS; ++i) z[i * NTHREADS + tid] = 0;
    if (tid < 4) sm.arrive[tid] = 0;
    if (tid == 4) sm.bad = 0;
    asm volatile("s_waitcnt lgkmcnt(0)" ::: "memory");
    __builtin_amdgcn_s_barrier();
  };
  u32 call;
  if (bid < NXP && wave < PWAVES) {
    const int peer = (rank + 1 + (bid >> 3)) & 7;
    const int slot = (bid & 7) * PWAVES + wave;
    u32x4 buf[XROUND][4];
    load_x_block(rx, peer, slot, lane, buf[0]);
    const u32 cnt = __hip_atomic_load(ctr, __ATOMIC_RELAXED, __HIP_MEMORY_SCOPE_AGENT);
#pragma unroll
    for (int r = 1; r < XROUND; ++r) load_x_block(rx, peer, r * NSLOT + slot, lane, buf[r]);
    FENCE();
    call = (u32)__builtin_amdgcn_readfirstlane(cnt) / (u32)NWG + 1;
    const auto rs = __builtin_amdgcn_make_buffer_rsrc(P.ws[peer], 0, (int)WS_NEEDED, 0x00020000);
    const u32 dbase = (u32)((call & 1) ? PAR_BYTES : 0) + (u32)rank * (u32)SRC_BYTES;
    const u32 fbase = (u32)FLAG_OFF + (u32)rank * (72 * 128);
#pragma unroll
    for (int r = 0; r <= XROUND; ++r) {
      if (r > 0) {
        const int n = (r - 1) * NSLOT + slot;
        WAIT_VM0();
        __builtin_amdgcn_raw_buffer_store_b32(call, rs, lane0, fbase + (u32)(64 + (n & 7)) * 128 + (u32)(n >> 3) * 4, AUX_SYS);
      }
      if (r < XROUND) {
        store_block<AUX_SYS>(buf[r], rs, lane * 16, dbase + (u32)(r * NSLOT + slot) * 4096);
      }
      FENCE();
      if (r == 0) {
        init_lds();
      }
    }
    STAMP(39);
  } else if (bid >= NXP && wave >= 4 && wave < 8) {
    const int n = (64 + (wave - 4) * NTILE + q) % NBLK;
    int row, c;
    decode_block(n, row, c);
    u32x4 o[4];
    u32 raw[16];
    if (n < 64) load_x_block(rx, rank, n, lane, o);
    else load_w_block(rw, row, c, lane, raw);
    const u32 cnt = __hip_atomic_load(ctr, __ATOMIC_RELAXED, __HIP_MEMORY_SCOPE_AGENT);
    FENCE();
    init_lds();
    call = (u32)__builtin_amdgcn_readfirstlane(cnt) / (u32)NWG + 1;
    if (n >= 64) tile_w_block(raw, o);
    const u32 dbase = (u32)((call & 1) ? PAR_BYTES : 0) + (u32)rank * (u32)SRC_BYTES;
    store_block<AUX_SYS>(o, rl, lane * 16, dbase + (u32)n * 4096);
    WAIT_VM0();
    if (lane == 0) {
      if (__hip_atomic_fetch_add(&sm.arrive[3], 1u, __ATOMIC_RELAXED, __HIP_MEMORY_SCOPE_WORKGROUP) == 3)
        __hip_atomic_fetch_add(tiled + xcd * 32, 1u, __ATOMIC_RELAXED, __HIP_MEMORY_SCOPE_AGENT);
      __builtin_amdgcn_raw_buffer_store_b32(call, rl, 0, (u32)FLAG_OFF + (u32)rank * (72 * 128) + (u32)row * 128 + (u32)c * 4, AUX_SYS);
    }
    STAMP(36);
  } else {
    const u32 cnt = __hip_atomic_load(ctr, __ATOMIC_RELAXED, __HIP_MEMORY_SCOPE_AGENT);
    FENCE();
    init_lds();
    call = (u32)__builtin_amdgcn_readfirstlane(cnt) / (u32)NWG + 1;
  }
  if (bid < 8) {
    __syncthreads();
    if (tid == 0) __hip_atomic_fetch_add(ctr, 1u, __ATOMIC_RELAXED, __HIP_MEMORY_SCOPE_AGENT);
    return;
  }
  const float scale = P.sx[0] * P.sw[0];
  const u32 sbase = (u32)((call & 1) ? PAR_BYTES : 0) + (u32)src * (u32)SRC_BYTES;
  u32* fsrc = (u32*)(myws + FLAG_OFF + (size_t)src * (72 * 128));
  const int nphase = gl < 2 ? 3 : 2;

#pragma unroll 1
  for (int t = 0; t < nphase; ++t) {
    const int grp = t < 2 ? t * 24 + xcd * 3 + gl : 48 + xcd * 2 + gl;
    i32x4 acc[4][4];
#pragma unroll
    for (int a = 0; a < 4; ++a)
#pragma unroll
      for (int b = 0; b < 4; ++b) acc[a][b] = i32x4{0, 0, 0, 0};
    int wfirst, wstep;
    w_index(grp, wfirst, wstep);
    u32* fptr = fsrc + (lane < 8 ? grp : 64 + m) * 32 + (lane & 7);
    u32 todo = 0xFF, ready = 0;
    u32x4 af[2][4], bf[2][4];
#pragma unroll
    for (int n = 0; n <= NCH; ++n) {
      if (n < NCH) {
        if (!ready) {
          ready = poll_ready(fptr, t == 0 ? 16 : 8, todo, lane, tmo, call);
          if (!ready) {
            LDS_STORE(&sm.bad, 1u);
            ready = todo;
          }
        }
        const int c = __builtin_ctz(ready);
        ready &= ready - 1;
        todo &= ~(1u << c);
        STAMP(t * 12 + n);
        const u32 xo = sbase + (u32)(c * 8 + m) * 4096;
        const u32 wo = __builtin_amdgcn_readfirstlane(sbase + (u32)(wfirst + c * wstep) * 4096);
#pragma unroll
        for (int j = 0; j < 4; ++j) bf[n & 1][j] = __builtin_amdgcn_raw_buffer_load_b128(rl, lane * 16 + j * 1024, wo, LD_AUX);
#pragma unroll
        for (int a = 0; a < 4; ++a) af[n & 1][a] = __builtin_amdgcn_raw_buffer_load_b128(rl, lane * 16 + a * 1024, xo, LD_AUX);
        __builtin_amdgcn_sched_barrier(0);
      }
      if (n > 0) {
#pragma unroll
        for (int a = 0; a < 4; ++a)
#pragma unroll
          for (int j = 0; j < 4; ++j)
            acc[a][j] = __builtin_amdgcn_mfma_i32_16x16x64_i8((i32x4)af[(n - 1) & 1][a], (i32x4)bf[(n - 1) & 1][j], acc[a][j], 0, 0, 0);
        __builtin_amdgcn_sched_barrier(0);
      }
    }
    STAMP(t * 12 + 8);
    int ln = lane;
    asm volatile("" : "+v"(ln));
#pragma unroll
    for (int a = 0; a < 4; ++a)
#pragma unroll
      for (int i = 0; i < 4; ++i)
#pragma unroll
        for (int jp = 0; jp < 2; ++jp) {
          const int lo = acc[a][2 * jp][i], hi = acc[a][2 * jp + 1][i];
          const u64 v = ((u64)(u32)(hi + (lo >> 31)) << 32) | (u32)lo;
          __hip_atomic_fetch_add(&sm.tile[t][a * 4 + i][jp][ln], v, __ATOMIC_RELAXED, __HIP_MEMORY_SCOPE_WORKGROUP);
        }
    int e0, e1;
    if (t + 1 == nphase) {
      __syncthreads();
      e0 = wave * 2; e1 = e0 + 2;
    } else {
      asm volatile("s_waitcnt lgkmcnt(0)" ::: "memory");
      u32 tk = 0;
      if (ln == 0) tk = __hip_atomic_fetch_add(&sm.arrive[t], 1u, __ATOMIC_RELAXED, __HIP_MEMORY_SCOPE_WORKGROUP);
      tk = __builtin_amdgcn_readfirstlane(tk);
      FENCE();
      e0 = 0; e1 = tk == 7 ? 16 : 0;
      if (t == 0 && tk == 7 && ln == 0) __hip_atomic_fetch_add(ctr, 1u, __ATOMIC_RELAXED, __HIP_MEMORY_SCOPE_AGENT);
    }
    STAMP(t * 12 + 9);
    if (e0 < e1) {
      const bool bad = LDS_LOAD(&sm.bad) != 0;
      const auto ro = __builtin_amdgcn_make_buffer_rsrc(P.out, 0, MS * GN * 4, 0x00020000);
      const int voff = (((ln >> 4) * 4) * GN + (ln & 15) * 4) * 4;
      for (int e = e0; e < e1; e += 2) {
#pragma unroll
        for (int h = 0; h < 2; ++h) {
          const u64 v0 = sm.tile[t][e + h][0][ln], v1 = sm.tile[t][e + h][1][ln];
          const int s0 = (int)(u32)v0, s2 = (int)(u32)v1;
          const int s1 = (int)(u32)(v0 >> 32) - (s0 >> 31), s3 = (int)(u32)(v1 >> 32) - (s2 >> 31);
          f32x4 y = f32x4{(float)s0 * scale, (float)s1 * scale, (float)s2 * scale, (float)s3 * scale};
          if (bad) y = f32x4{__builtin_nanf(""), __builtin_nanf(""), __builtin_nanf(""), __builtin_nanf("")};
          const int row = m * 64 + ((e + h) >> 2) * 16 + ((e + h) & 3);
          store16<OUT_AUX>(__builtin_bit_cast(u32x4, y), ro, voff, (row * GN + grp * 64) * 4);
        }
      }
    }
    STAMP(t * 12 + 10);
  }
}

}

extern "C" void kernel_launch(int rank, int nranks, ncclComm_t comm,
                              hipStream_t stream,
                              void* const* d_in, const int* in_sizes, int n_in,
                              void* d_out, int out_size,
                              const int* in_dtypes, int out_dtype,
                              void* const* d_out_all,
                              void* const* d_ws_all, size_t ws_bytes) {
  if (nranks != NRANK || n_in != 4 || ws_bytes < WS_NEEDED || out_size != MS * GN ||
      in_sizes[0] != GM * KS || in_sizes[1] != KS * GN || (unsigned)rank >= (unsigned)NRANK) {
    fprintf(stderr, "kernel_launch: unexpected problem shape\n");
    return;
  }
  Params p;
  for (int i = 0; i < NRANK; ++i) p.ws[i] = (char*)d_ws_all[i];
  p.x = (const int8_t*)d_in[0];
  p.w = (const int8_t*)d_in[1];
  p.sx = (const float*)d_in[2];
  p.sw = (const float*)d_in[3];
  p.out = (float*)d_out;
  p.rank = rank;
  hipLaunchKernelGGL(gemm_gather_kernel, dim3(NWG), dim3(NTHREADS), 0, stream, p);
  HIP_CHECK(hipGetLastError());
}
```

```cpp
#include <hip/hip_runtime.h>
#include <rccl/rccl.h>
#include <cstdio>
#include <cstdint>

#define HIP_CHECK(expr) do { hipError_t _e = (expr); if (_e != hipSuccess) { \
    fprintf(stderr, "HIP error %s at %s:%d\n", hipGetErrorString(_e), __FILE__, __LINE__); \
    return; } } while (0)

namespace {

typedef unsigned int u32;
typedef unsigned long long u64;
typedef __attribute__((ext_vector_type(4))) int i32x4;
typedef __attribute__((ext_vector_type(4))) unsigned u32x4;
typedef __attribute__((ext_vector_type(4))) float f32x4;

constexpr int NRANK = 8;
constexpr int GM = 4096, GN = 4096, GK = 4096;
constexpr int MS = GM / NRANK;
constexpr int KS = GK / NRANK;
constexpr int NTHREADS = 512;
constexpr int NWG = 256;
constexpr int NXP = 56;
constexpr int NTILE = 144;
constexpr int PULL0 = NXP + NTILE;
constexpr int NCH = 8;
constexpr int AUX_SYS = 17;
constexpr int OOB = 0x7FFF0000;
constexpr u64 PULL_GATE = 500;
constexpr u64 DEADLINE_TICKS = 20000000ull;

#define NSH1 44
constexpr int NSH = NSH1 + 64;
constexpr int KP = NSH / 4;
constexpr u32 NL_IN = 64 + 288 + NSH;
constexpr u32 NL_CR = KP + 64 + 288 - NSH + 224;
constexpr u32 NLIST = NL_IN > NL_CR ? NL_IN : NL_CR;
static_assert(NSH1 % 4 == 0 && NLIST < NTHREADS, "pull lists");

constexpr u32 SRC_BYTES = 640 * 4096;
constexpr u32 XST_OFF = 8 * SRC_BYTES;
constexpr u32 OBX_OFF = XST_OFF + 512 * 4096;
constexpr u32 PAR_BYTES = OBX_OFF + 4 * 224 * 4096;
constexpr u32 CTRL_OFF = 2 * PAR_BYTES;
constexpr u32 TMO_OFF = CTRL_OFF + 256;
constexpr u32 TILED_OFF = CTRL_OFF + 512;
constexpr u32 RDY_OFF = CTRL_OFF + 2048;
constexpr u32 QUEUE_OFF = CTRL_OFF + 3072;
constexpr u32 PCNT_OFF = CTRL_OFF + 4096;
constexpr u32 NEED_OFF = CTRL_OFF + 5120;
constexpr u32 DUMMY_OFF = CTRL_OFF + 8192;
constexpr u32 FLAG_OFF = CTRL_OFF + 16384;
constexpr u32 FLAG_SRC = 80 * 128;
constexpr u32 WS_NEEDED = FLAG_OFF + NRANK * FLAG_SRC;

enum { K_X = 0, K_W = 1, K_P = 2, K_R = 3, K_O = 4 };
struct Lists { unsigned short v[2][NLIST]; unsigned short sh[NSH]; };
constexpr Lists make_lists() {
  Lists L{};
  bool shared[288] = {};
  for (int i = 0; i < NSH1; ++i) L.sh[i] = 96 + (i * 128 + 64) / NSH1;
  for (int i = 0; i < 64; ++i) L.sh[NSH1 + i] = 224 + i;
  for (int i = 0; i < NSH; ++i) shared[L.sh[i]] = true;
  for (int cr = 0; cr < 2; ++cr) {
    int n = 0, k = 0;
    if (cr) for (int i = 0; i < KP; ++i) L.v[1][n++] = K_P << 12 | i;
    for (int w = 0; w < 288; ++w) {
      if (w < 48 && w % 6 == 0) for (int i = 0; i < 8; ++i) L.v[cr][n++] = K_X << 12 | (w / 6 * 8 + i);
      if (!cr && k < KP && w >= L.sh[4 * k]) { for (int j = 0; j < 4; ++j) L.v[0][n++] = K_R << 12 | (j * 64 + k); ++k; }
      if (!cr || !shared[w]) L.v[cr][n++] = K_W << 12 | w;
    }
    if (cr) for (int i = 0; i < 224; ++i) L.v[1][n++] = K_O << 12 | i;
  }
  return L;
}
constexpr Lists LISTS = make_lists();

struct Params {
  char* ws[NRANK];
  const int8_t* x;
  const int8_t* w;
  const float* sx;
  const float* sw;
  float* out;
  int rank;
};

#define FENCE() asm volatile("" ::: "memory")
#define WAIT_VM(n) do { FENCE(); __builtin_amdgcn_s_waitcnt(0x0F70 | (n)); FENCE(); } while (0)
#define LDS_LOAD(p) __hip_atomic_load((p), __ATOMIC_RELAXED, __HIP_MEMORY_SCOPE_WORKGROUP)
#define LDS_STORE(p, v) __hip_atomic_store((p), (v), __ATOMIC_RELAXED, __HIP_MEMORY_SCOPE_WORKGROUP)
#define LDS_ADD(p, v) __hip_atomic_fetch_add((p), (v), __ATOMIC_RELAXED, __HIP_MEMORY_SCOPE_WORKGROUP)
#define GPU_LOAD(p) __hip_atomic_load((p), __ATOMIC_RELAXED, __HIP_MEMORY_SCOPE_AGENT)
#define GPU_ADD(p, v) __hip_atomic_fetch_add((p), (v), __ATOMIC_RELAXED, __HIP_MEMORY_SCOPE_AGENT)
#define SYS_LOAD(p) __hip_atomic_load((p), __ATOMIC_RELAXED, __HIP_MEMORY_SCOPE_SYSTEM)
#define MAKE_RSRC(p, bytes) __builtin_amdgcn_make_buffer_rsrc((void*)(p), 0, (int)(bytes), 0x00020000)

template <class R>
__device__ __forceinline__ void store16(u32x4 d, R r, int voff, u32 soff) {
  __builtin_amdgcn_raw_buffer_store_b128(d, r, voff, soff, AUX_SYS);
  asm volatile("s_nop 1" ::"v"(d) : "memory");
}
template <class R>
__device__ __forceinline__ void store_block(const u32x4 (&d)[4], R r, int voff, u32 soff, int stride = 1024) {
#pragma unroll
  for (int j = 0; j < 4; ++j) __builtin_amdgcn_raw_buffer_store_b128(d[j], r, voff, soff + j * stride, AUX_SYS);
  asm volatile("s_nop 1" ::"v"(d[0]), "v"(d[1]), "v"(d[2]), "v"(d[3]) : "memory");
}

__device__ __forceinline__ void decode_need(int w, int& lg, int& c) {
  if (w < 192) { const int sp = w / 48, r = w - sp * 48; c = r / 6; lg = sp * 6 + r - c * 6; }
  else if (w < 224) { const int r = w - 192; c = r >> 2; lg = 24 + (r & 3); }
  else { const int r = w - 224; c = r >> 3; lg = 28 + (r & 7); }
}
__device__ __forceinline__ int global_group(int h, int lg) { return lg < 28 ? 28 * h + lg : 28 + lg; }

template <class R>
__device__ __forceinline__ void load_x_block(R rx, int gm, int c, int lane, u32x4 (&o)[4]) {
#pragma unroll
  for (int a = 0; a < 4; ++a)
    o[a] = __builtin_amdgcn_raw_buffer_load_b128(rx, (lane & 15) * KS + (lane >> 4) * 16, (gm * 64 + a * 16) * KS + c * 64, 0);
}
template <class R>
__device__ __forceinline__ void load_w_block(R rw, int grp, int c, int lane, u32 (&raw)[16]) {
#pragma unroll
  for (int i = 0; i < 16; ++i)
    raw[i] = __builtin_amdgcn_raw_buffer_load_b32(rw, (lane >> 4) * 16 * GN + (lane & 15) * 4, (c * 64 + i) * GN + grp * 64, 0);
}
__device__ __forceinline__ void tile_w_block(const u32 (&raw)[16], u32x4 (&o)[4]) {
#pragma unroll
  for (int k = 0; k < 4; ++k) {
    const u32 t0 = __builtin_amdgcn_perm(raw[4 * k + 1], raw[4 * k], 0x05010400u);
    const u32 t1 = __builtin_amdgcn_perm(raw[4 * k + 1], raw[4 * k], 0x07030602u);
    const u32 t2 = __builtin_amdgcn_perm(raw[4 * k + 3], raw[4 * k + 2], 0x05010400u);
    const u32 t3 = __builtin_amdgcn_perm(raw[4 * k + 3], raw[4 * k + 2], 0x07030602u);
    o[0][k] = __builtin_amdgcn_perm(t2, t0, 0x05040100u);
    o[1][k] = __builtin_amdgcn_perm(t2, t0, 0x07060302u);
    o[2][k] = __builtin_amdgcn_perm(t3, t1, 0x05040100u);
    o[3][k] = __builtin_amdgcn_perm(t3, t1, 0x07060302u);
  }
}

struct Smem {
  union {
    u64 tile[3][16][2][64];
    u32x4 table[NLIST + 1];
    char fill[96 * 1024];
  };
  u32 arrive[8];
  u32 bad;
  u32 pad[3];
};

__device__ __forceinline__ bool deadline_hit(u32 spins, u64& t0, u32* tmo, u32 call) {
  if (spins == 0) t0 = __builtin_amdgcn_s_memrealtime();
  if ((spins & 63) != 63) return false;
  if (GPU_LOAD(tmo) == call || __builtin_amdgcn_s_memrealtime() - t0 > DEADLINE_TICKS) {
    __hip_atomic_store(tmo, call, __ATOMIC_RELAXED, __HIP_MEMORY_SCOPE_AGENT);
    return true;
  }
  return false;
}

__device__ __forceinline__ u32 poll_ready(u32* fptr, u32 todo, int lane, u32* tmo, u32 call) {
  u64 t0 = 0;
  for (u32 spins = 0;; ++spins) {
    u32 f = call;
    if (lane < 16) f = SYS_LOAD(fptr);
    const u64 b = __ballot((int)(f - call) >= 0);
    const u32 ready = (u32)(b & (b >> 8)) & todo;
    if (ready) { FENCE(); return ready; }
    if (deadline_hit(spins, t0, tmo, call)) return 0;
    __builtin_amdgcn_s_sleep(4);
  }
}

__global__ void __launch_bounds__(NTHREADS) gemm_gather_kernel(Params P) {
  __shared__ __attribute__((aligned(16))) Smem sm;
  const int tid = threadIdx.x;
  const int lane = tid & 63;
  const int wave = __builtin_amdgcn_readfirstlane(tid >> 6);
  const int bid = blockIdx.x;
  const int rank = P.rank;
  const int h = rank & 1, pos = rank >> 1;
  char* myws = P.ws[rank];
  u32* ctr = (u32*)(myws + CTRL_OFF);
  u32* tmo = (u32*)(myws + TMO_OFF);
  u32* tiled = (u32*)(myws + TILED_OFF);
  const auto rx = MAKE_RSRC(P.x, GM * KS);
  const auto rw = MAKE_RSRC(P.w, KS * GN);
  const auto rl = MAKE_RSRC(myws, WS_NEEDED);
  const int lane0 = lane == 0 ? 0 : OOB;

  if (bid >= PULL0) {
    const u64 tstart = __builtin_amdgcn_s_memrealtime();
    const u32 cnt = GPU_LOAD(ctr);
    const int link = (bid - PULL0) >> 3;
    const int peer = (rank + 1 + link) & 7;
    const int cross = (peer ^ rank) & 1;
    const u32 dummy = DUMMY_OFF + (bid - PULL0) * 128;
    const u32 nlist = cross ? NL_CR : NL_IN;
    if (tid <= (int)NLIST) {
      u32x4 e = {0, 0, dummy, 63 << 4};
      if (tid < (int)nlist) {
        const u32 code = LISTS.v[cross][tid], kind = code >> 12, p = code & 4095;
        if (kind == K_X) {
          const int c = p >> 3, hr = p & 7;
          const int gm = (2 * (hr >> 1) + 1 - h) * 8 + 2 * pos + (hr & 1);
          e[0] = XST_OFF + (c * 64 + gm) * 4096;
          e[1] = peer * SRC_BYTES + (c * 16 + 8 + hr) * 4096;
          e[2] = FLAG_OFF + peer * FLAG_SRC + (64 + 8 + hr) * 128 + c * 4;
          e[3] = 1 | 63 << 4;
        } else if (kind == K_O) {
          const int lg = p >> 3, sub = (p >> 2) & 1, mi = p & 3;
          e[0] = OBX_OFF + (pos * 224 + p) * 4096;
          e[1] = (((2 * (peer >> 1) + sub) * 64 + mi * 16) * GN + (28 * (1 - h) + lg) * 64) * 4;
          e[3] = 1 | 2 | (p >> 2) << 4;
        } else {
          int w = p, s = peer;
          u32 meta = 1 | 63 << 4;
          if (kind != K_W) {
            w = LISTS.sh[4 * (p & 63) + (kind == K_P ? pos : peer >> 1)];
            if (kind == K_P) meta |= 4;
            else { s = 2 * (p >> 6) + 1 - h; meta = 1 | (56 + (p >> 6)) << 4; }
          }
          int lg, c;
          decode_need(w, lg, c);
          const int G = global_group(h, lg);
          e[0] = e[1] = s * SRC_BYTES + (128 + c * 64 + G) * 4096;
          e[2] = FLAG_OFF + s * FLAG_SRC + G * 128 + c * 4;
          e[3] = meta;
        }
      }
      sm.table[tid] = e;
    }
    const u32 call = (u32)__builtin_amdgcn_readfirstlane(cnt) / (u32)NWG + 1;
    __syncthreads();
    if (wave == 0) {
      while (__builtin_amdgcn_s_memrealtime() - tstart < PULL_GATE) __builtin_amdgcn_s_sleep(4);
      const auto rs = MAKE_RSRC(P.ws[peer], WS_NEEDED);
      const u32 pbase = (call & 1) ? PAR_BYTES : 0;
      bool failed = false;
      {
        u32* rdy = (u32*)(myws + RDY_OFF) + peer * 32;
        u64 t0 = 0;
        for (u32 spins = 0; (int)(SYS_LOAD(rdy) - call) < 0; ++spins) {
          if (deadline_hit(spins, t0, tmo, call)) { failed = true; break; }
          __builtin_amdgcn_s_sleep(2);
        }
        FENCE();
      }
      if (lane == 0) GPU_ADD(ctr, 1u);
      struct Ent { u32 so, dof, fo, meta; };
      auto lookup = [&](u32 i) __attribute__((always_inline)) -> Ent {
        const u32x4 e = sm.table[i < NLIST ? i : NLIST];
        return Ent{(u32)__builtin_amdgcn_readfirstlane(e[0]), (u32)__builtin_amdgcn_readfirstlane(e[1]),
                   (u32)__builtin_amdgcn_readfirstlane(e[2]), (u32)__builtin_amdgcn_readfirstlane(e[3])};
      };
      auto fetch = [&](const Ent& e, u32x4 (&d)[4]) __attribute__((always_inline)) {
        const int vo = (e.meta & 1) ? lane * 16 : OOB;
#pragma unroll
        for (int j = 0; j < 4; ++j) d[j] = __builtin_amdgcn_raw_buffer_load_b128(rs, vo + j * 1024, pbase + e.so, AUX_SYS);
        FENCE();
      };
      const int ovoff = (((lane >> 4) * 4) * GN + (lane & 15) * 4) * 4;
      auto put = [&](const Ent& e, u32x4 (&d)[4]) __attribute__((always_inline)) {
        const bool o = e.meta & 2;
        const u32 nanm = failed ? 0x7FC00000u : 0u;
#pragma unroll
        for (int j = 0; j < 4; ++j) d[j] |= u32x4{nanm, nanm, nanm, nanm};
        const auto rd = MAKE_RSRC(o ? (void*)P.out : (void*)myws, o ? MS * GN * 4 : WS_NEEDED);
        const int vo = (e.meta & 1) ? (o ? ovoff : lane * 16) : OOB;
        store_block(d, rd, vo, o ? e.dof : pbase + e.dof, o ? GN * 4 : 1024);
      };
      auto flag = [&](const Ent& e) __attribute__((always_inline)) {
        __builtin_amdgcn_raw_buffer_store_b32(call, rl, lane0, failed ? dummy : e.fo, AUX_SYS);
        FENCE();
        if ((e.meta & 4) && lane == 0) GPU_ADD((u32*)(myws + PCNT_OFF) + link * 32, 1u);
        FENCE();
      };
      auto ticket = [&](bool v) __attribute__((always_inline)) -> u32 {
        const u32 tk = (u32)__builtin_amdgcn_raw_ptr_buffer_atomic_add_i32(1, rl, v ? lane0 : OOB,
                                                                           QUEUE_OFF + ((call & 1) * 8 + link) * 64, 0);
        FENCE();
        return tk;
      };
      auto peek = [&]() __attribute__((always_inline)) -> u32 {
        const u32 f = __builtin_amdgcn_raw_buffer_load_b32(rl, lane * 4, NEED_OFF + peer * 256, AUX_SYS);
        FENCE();
        return f;
      };
      u32x4 d0[4], d1[4];
      Ent e0 = lookup(bid & 7), e1 = lookup((bid & 7) + 8), e2 = lookup((bid & 7) + 16), e3 = lookup((bid & 7) + 24);
      fetch(e0, d0);
      fetch(e1, d1);
      WAIT_VM(4);  put(e0, d0); u32 tk0 = ticket(true), rd0 = peek(); fetch(e2, d0);
      WAIT_VM(10); put(e1, d1); u32 tk1 = ticket(true), rd1 = peek(); fetch(e3, d1);
      u64 known = failed ? ~0ull : 1ull << 63, tw = 0;
      u32 p0 = NLIST, p1 = NLIST, idle = 0;
      bool has0 = true, has1 = true, ended = false, full = false;
#define PULL_STEP(D, TK, RD, HAS, OHAS, CNT) {                                                           \
        if (!full) WAIT_VM(CNT); else WAIT_VM(0);                                                        \
        if (HAS) {                                                                                       \
          const u32 i = (u32)__builtin_amdgcn_readfirstlane(TK) + 32;                                    \
          if (i >= nlist) ended = true; else if (p0 >= NLIST) p0 = i; else p1 = i;                       \
        }                                                                                                \
        Ent e4 = lookup(p0);                                                                             \
        known |= __ballot((int)(RD - call) >= 0);                                                        \
        put(e2, D);                                                                                      \
        if ((known >> (e4.meta >> 4)) & 1) { p0 = p1; p1 = NLIST; }                                      \
        else {                                                                                           \
          e4.meta = 63 << 4; e4.fo = dummy;                                                              \
          if (deadline_hit(idle++, tw, tmo, call)) { failed = true; known = ~0ull; }                     \
        }                                                                                                \
        HAS = !ended && (p0 < NLIST) + (p1 < NLIST) + OHAS < 2;                                          \
        TK = ticket(HAS);                                                                                \
        RD = peek();                                                                                     \
        fetch(e4, D);                                                                                    \
        flag(e0);                                                                                        \
        if (full) { if (e1.meta & 1) flag(e1); e1.meta = 0; e1.fo = dummy; }                             \
        full = !((e2.meta & e4.meta & 1) && HAS);                                                        \
        if (e2.meta & 2) e2.meta = 0;                             \
        e0 = e1; e1 = e2; e2 = e3; e3 = e4;                                                              \
      }
      PULL_STEP(d0, tk0, rd0, has0, has1, 10)
      PULL_STEP(d1, tk1, rd1, has1, has0, 11)
#pragma unroll 1
      do {
        PULL_STEP(d0, tk0, rd0, has0, has1, 12)
        PULL_STEP(d1, tk1, rd1, has1, has0, 12)
      } while (((e0.meta | e1.meta | e2.meta | e3.meta) & 1) || has0 || has1 || p0 < NLIST);
#undef PULL_STEP
    } else if (wave == 2 && (bid & 7) == 0) {
      __builtin_amdgcn_raw_buffer_store_b32(0, rl, lane0, QUEUE_OFF + ((~call & 1) * 8 + link) * 64, AUX_SYS);
    } else if (wave == 3 && (bid & 7) == 1 && cross) {
      u32* pc = (u32*)(myws + PCNT_OFF) + link * 32;
      u64 t0 = 0;
      bool ok = true;
      while (__builtin_amdgcn_s_memrealtime() - tstart < PULL_GATE + 200) __builtin_amdgcn_s_sleep(16);
      for (u32 spins = 0; (int)(GPU_LOAD(pc) - call * KP) < 0; ++spins) {
        if (deadline_hit(spins, t0, tmo, call)) { ok = false; break; }
        __builtin_amdgcn_s_sleep(8);
      }
      FENCE();
      if (ok && GPU_LOAD(tmo) != call) {
#pragma unroll
        for (int u = 1; u < 4; ++u) {
          const auto ro = MAKE_RSRC(P.ws[(rank + 2 * u) & 7], WS_NEEDED);
          __builtin_amdgcn_raw_buffer_store_b32(call, ro, lane0, NEED_OFF + rank * 256 + (56 + (peer >> 1)) * 4, AUX_SYS);
        }
      }
    } else if (wave == 1 && bid == PULL0) {
      const u32 target = call * (NTILE / 8);
      u64 t0 = 0;
      bool ok = true;
      for (u32 spins = 0;; ++spins) {
        u32 f = target;
        if (lane < 8) f = GPU_LOAD(tiled + lane * 32);
        if (__all((int)(f - target) >= 0)) break;
        if (deadline_hit(spins, t0, tmo, call)) { ok = false; break; }
        __builtin_amdgcn_s_sleep(1);
      }
      FENCE();
      if (ok) {
#pragma unroll
        for (int i = 1; i < NRANK; ++i) {
          const auto rs = MAKE_RSRC(P.ws[(rank + i) & 7], WS_NEEDED);
          __builtin_amdgcn_raw_buffer_store_b32(call, rs, lane0, RDY_OFF + rank * 128, AUX_SYS);
        }
      }
    }
    return;
  }

  const int q = bid < NXP ? NTILE + bid - 8 : bid - NXP;
  const int xcd = bid & 7;
  const int idx = q >> 3;
  const int src = (rank - wave) & 7;
  auto init_lds = [&]() __attribute__((always_inline)) {
    u64* z = &sm.tile[0][0][0][0];
#pragma unroll
    for (int i = 0; i < 3 * 16 * 2 * 64 / NTHREADS; ++i) z[i * NTHREADS + tid] = 0;
    if (tid < 8) sm.arrive[tid] = 0;
    if (tid == 8) sm.bad = 0;
    asm volatile("s_waitcnt lgkmcnt(0)" ::: "memory");
    __builtin_amdgcn_s_barrier();
  };
  u32 call;
  if (bid < NXP && wave < 4) {
    const int peer = (rank + 1 + (bid >> 3)) & 7;
    const int slot = (bid & 7) * 4 + wave;
    u32x4 b0[4], b1[4];
    load_x_block(rx, peer * 8 + (slot & 7), slot >> 3, lane, b0);
    const u32 cnt = GPU_LOAD(ctr);
    load_x_block(rx, peer * 8 + (slot & 7), 4 + (slot >> 3), lane, b1);
    FENCE();
    call = (u32)__builtin_amdgcn_readfirstlane(cnt) / (u32)NWG + 1;
    const auto rs = MAKE_RSRC(P.ws[peer], WS_NEEDED);
    const u32 doff = ((call & 1) ? PAR_BYTES : 0) + (u32)rank * SRC_BYTES + (u32)((slot >> 3) * 16 + (slot & 7)) * 4096;
    const u32 foff = FLAG_OFF + (u32)rank * FLAG_SRC + (u32)(64 + (slot & 7)) * 128 + (u32)(slot >> 3) * 4;
    store_block(b0, rs, lane * 16, doff);
    init_lds();
    WAIT_VM(0);
    __builtin_amdgcn_raw_buffer_store_b32(call, rs, lane0, foff, AUX_SYS);
    store_block(b1, rs, lane * 16, doff + 64 * 4096);
    WAIT_VM(0);
    __builtin_amdgcn_raw_buffer_store_b32(call, rs, lane0, foff + 16, AUX_SYS);
    FENCE();
  } else if (bid >= NXP) {
    const int n = wave * NTILE + q;
    const bool isw = n < 512;
    const int a = n & 63, c = (n >> 6) & 7;
    u32x4 o[4];
    u32 raw[16];
    if (isw) load_w_block(rw, a, c, lane, raw);
    else load_x_block(rx, a, c, lane, o);
    const u32 cnt = GPU_LOAD(ctr);
    FENCE();
    init_lds();
    call = (u32)__builtin_amdgcn_readfirstlane(cnt) / (u32)NWG + 1;
    const u32 pbase = (call & 1) ? PAR_BYTES : 0;
    const u32 dbase = pbase + (u32)rank * SRC_BYTES;
    int frow = -1;
    if (isw) {
      tile_w_block(raw, o);
      store_block(o, rl, lane * 16, dbase + (u32)(128 + n) * 4096);
      frow = a;
    } else if (n < 1024) {
      const int owner = a >> 3, mt = a & 7;
      const bool mine = owner == rank, helped = ((owner ^ rank) & 1) && (mt >> 1) == pos;
      if (!helped) store_block(o, rl, lane * 16, pbase + XST_OFF + (u32)(n - 512) * 4096);
      if (mine || helped) {
        const int rt = mine ? mt : 8 + (owner >> 1) * 2 + (mt & 1);
        store_block(o, rl, lane * 16, dbase + (u32)(c * 16 + rt) * 4096);
        frow = 64 + rt;
      }
    }
    WAIT_VM(0);
    if (lane == 0 && LDS_ADD(&sm.arrive[3], 1u) == 7) GPU_ADD(tiled + xcd * 32, 1u);
    if (frow >= 0)
      __builtin_amdgcn_raw_buffer_store_b32(call, rl, lane0, FLAG_OFF + (u32)rank * FLAG_SRC + (u32)frow * 128 + (u32)c * 4, AUX_SYS);
    FENCE();
  } else {
    const u32 cnt = GPU_LOAD(ctr);
    FENCE();
    init_lds();
    call = (u32)__builtin_amdgcn_readfirstlane(cnt) / (u32)NWG + 1;
  }
  if (bid < 8) {
    __syncthreads();
    if (tid == 0) GPU_ADD(ctr, 1u);
    return;
  }

  const float scale = P.sx[0] * P.sw[0];
  const u32 pbase = (call & 1) ? PAR_BYTES : 0;
  const u32 sbase = pbase + (u32)src * SRC_BYTES;
  u32* fsrc = (u32*)(myws + FLAG_OFF + (u32)src * FLAG_SRC);
  const int nphase = idx < 16 ? 3 : 2;

#pragma unroll 1
  for (int t = 0; t < nphase; ++t) {
    const bool tail = t == 2 && idx >= 8;
    const int lg = tail ? 28 + xcd : t * 12 + 2 * (idx >> 2) + (xcd >> 2);
    const int rt = tail ? idx - 8 : (xcd & 3) * 4 + (idx & 3);
    const int G = global_group(h, lg);
    i32x4 acc[4][4];
#pragma unroll
    for (int a = 0; a < 4; ++a)
#pragma unroll
      for (int b = 0; b < 4; ++b) acc[a][b] = i32x4{0, 0, 0, 0};
    u32* fptr = fsrc + (lane < 8 ? G : 64 + rt) * 32 + (lane & 7);
    u32 todo = 0xFF, ready = 0;
    u32x4 af[2][4], bf[2][4];
#pragma unroll
    for (int n = 0; n <= NCH; ++n) {
      if (n < NCH) {
        if (!ready) {
          ready = poll_ready(fptr, todo, lane, tmo, call);
          if (!ready) {
            LDS_STORE(&sm.bad, 1u);
            ready = todo;
          }
        }
        const int c = __builtin_ctz(ready);
        ready &= ready - 1;
        todo &= ~(1u << c);
        const u32 xo = sbase + (u32)(c * 16 + rt) * 4096;
        const u32 wo = sbase + (u32)(128 + c * 64 + G) * 4096;
#pragma unroll
        for (int j = 0; j < 4; ++j) bf[n & 1][j] = __builtin_amdgcn_raw_buffer_load_b128(rl, lane * 16 + j * 1024, wo, AUX_SYS);
#pragma unroll
        for (int a = 0; a < 4; ++a) af[n & 1][a] = __builtin_amdgcn_raw_buffer_load_b128(rl, lane * 16 + a * 1024, xo, AUX_SYS);
        __builtin_amdgcn_sched_barrier(0);
      }
      if (n > 0) {
#pragma unroll
        for (int a = 0; a < 4; ++a)
#pragma unroll
          for (int j = 0; j < 4; ++j)
            acc[a][j] = __builtin_amdgcn_mfma_i32_16x16x64_i8((i32x4)af[(n - 1) & 1][a], (i32x4)bf[(n - 1) & 1][j], acc[a][j], 0, 0, 0);
        __builtin_amdgcn_sched_barrier(0);
      }
    }
    int ln = lane;
    asm volatile("" : "+v"(ln));
#pragma unroll
    for (int a = 0; a < 4; ++a)
#pragma unroll
      for (int i = 0; i < 4; ++i)
#pragma unroll
        for (int jp = 0; jp < 2; ++jp) {
          const int lo = acc[a][2 * jp][i], hi = acc[a][2 * jp + 1][i];
          LDS_ADD(&sm.tile[t][a * 4 + i][jp][ln], ((u64)(u32)(hi + (lo >> 31)) << 32) | (u32)lo);
        }
    int e0, e1;
    const bool last = t + 1 == nphase;
    if (last) {
      __syncthreads();
      e0 = wave * 4; e1 = wave < 4 ? e0 + 4 : e0;
    } else {
      asm volatile("s_waitcnt lgkmcnt(0)" ::: "memory");
      u32 tk = 0;
      if (ln == 0) tk = LDS_ADD(&sm.arrive[t], 1u);
      tk = __builtin_amdgcn_readfirstlane(tk);
      FENCE();
      e0 = 0; e1 = tk == 7 ? 16 : 0;
      if (t == 0 && tk == 7 && ln == 0) GPU_ADD(ctr, 1u);
    }
    if (e0 < e1) {
      const bool bad = LDS_LOAD(&sm.bad) != 0;
      const bool own = rt < 8;
      const int op = (rt - 8) >> 1;
      const auto ro = MAKE_RSRC(own ? (void*)P.out : (void*)myws, own ? MS * GN * 4 : WS_NEEDED);
      const int voff = own ? (((ln >> 4) * 4) * GN + (ln & 15) * 4) * 4 : ln * 16;
      const u32 obx = pbase + OBX_OFF + (u32)(op * 224 + (lg * 2 + (rt & 1)) * 4) * 4096;
      for (int e = e0; e < e1; e += 4) {
        u64 v[4][2];
#pragma unroll
        for (int k = 0; k < 4; ++k) { v[k][0] = sm.tile[t][e + k][0][ln]; v[k][1] = sm.tile[t][e + k][1][ln]; }
#pragma unroll
        for (int k = 0; k < 4; ++k) {
          const int s0 = (int)(u32)v[k][0], s2 = (int)(u32)v[k][1];
          const int s1 = (int)(u32)(v[k][0] >> 32) - (s0 >> 31), s3 = (int)(u32)(v[k][1] >> 32) - (s2 >> 31);
          f32x4 y = f32x4{(float)s0 * scale, (float)s1 * scale, (float)s2 * scale, (float)s3 * scale};
          if (bad) y = f32x4{__builtin_nanf(""), __builtin_nanf(""), __builtin_nanf(""), __builtin_nanf("")};
          const u32 so = own ? (u32)((rt * 64 + e * 4 + k) * GN + G * 64) * 4 : obx + (u32)(e + k) * 1024;
          store16(__builtin_bit_cast(u32x4, y), ro, voff, so);
        }
      }
      if (!own) {
        WAIT_VM(0);
        u32 n = 3;
        if (last && ln == 0) n = LDS_ADD(&sm.arrive[4], 1u);
        if (__builtin_amdgcn_readfirstlane(n) == 3) {
          const auto rs = MAKE_RSRC(P.ws[(2 * op + 1 - h) & 7], WS_NEEDED);
          __builtin_amdgcn_raw_buffer_store_b32(call, rs, lane0, NEED_OFF + rank * 256 + (lg * 2 + (rt & 1)) * 4, AUX_SYS);
        }
      }
    }
  }
}

}

extern "C" void kernel_launch(int rank, int nranks, ncclComm_t comm,
                              hipStream_t stream,
                              void* const* d_in, const int* in_sizes, int n_in,
                              void* d_out, int out_size,
                              const int* in_dtypes, int out_dtype,
                              void* const* d_out_all,
                              void* const* d_ws_all, size_t ws_bytes) {
  if (nranks != NRANK || n_in != 4 || ws_bytes < WS_NEEDED || out_size != MS * GN ||
      in_sizes[0] != GM * KS || in_sizes[1] != KS * GN || (unsigned)rank >= (unsigned)NRANK) {
    fprintf(stderr, "kernel_launch: unexpected problem shape\n");
    return;
  }
  Params p;
  for (int i = 0; i < NRANK; ++i) p.ws[i] = (char*)d_ws_all[i];
  p.x = (const int8_t*)d_in[0];
  p.w = (const int8_t*)d_in[1];
  p.sx = (const float*)d_in[2];
  p.sw = (const float*)d_in[3];
  p.out = (float*)d_out;
  p.rank = rank;
  hipLaunchKernelGGL(gemm_gather_kernel, dim3(NWG), dim3(NTHREADS), 0, stream, p);
  HIP_CHECK(hipGetLastError());
}
```

```cpp
#include <hip/hip_runtime.h>
#include <rccl/rccl.h>
#include <cstdio>
#include <cstdint>

#define HIP_CHECK(expr) do { hipError_t _e = (expr); if (_e != hipSuccess) { \
    fprintf(stderr, "HIP error %s at %s:%d\n", hipGetErrorString(_e), __FILE__, __LINE__); \
    return; } } while (0)

namespace {

typedef unsigned int u32;
typedef unsigned long long u64;
typedef __attribute__((ext_vector_type(4))) int i32x4;
typedef __attribute__((ext_vector_type(4))) unsigned u32x4;
typedef __attribute__((ext_vector_type(4))) float f32x4;

constexpr int NRANK = 8;
constexpr int GM = 4096, GN = 4096, GK = 4096;
constexpr int MS = GM / NRANK;
constexpr int KS = GK / NRANK;
constexpr int NTHREADS = 512;
constexpr int NWG = 256;
constexpr int NXP = 56;
constexpr int NTILE = 144;
constexpr int PULL0 = NXP + NTILE;
constexpr int NCH = 8;
constexpr int AUX_SYS = 17;
constexpr int OOB = 0x7FFF0000;
constexpr u64 PULL_GATE = 550;
constexpr u64 DEADLINE_TICKS = 100000000ull;

constexpr int NSH1 = 44;
constexpr int NSH = NSH1 + 64;
constexpr int KP = NSH / 4;
constexpr int KP1 = NSH1 / 4;
constexpr int LATEP = 96;
constexpr u32 NL_IN = 64 + 288 + NSH;
constexpr u32 NL_CR = KP + 64 + 288 - NSH + 224;
constexpr u32 NLIST = NL_IN > NL_CR ? NL_IN : NL_CR;
static_assert(NSH1 % 4 == 0 && NLIST < NTHREADS, "pull lists");

constexpr u32 SRC_BYTES = 640 * 4096;
constexpr u32 XST_OFF = 8 * SRC_BYTES;
constexpr u32 OBX_OFF = XST_OFF + 512 * 4096;
constexpr u32 PAR_BYTES = OBX_OFF + 4 * 224 * 4096;
constexpr u32 CTRL_OFF = 2 * PAR_BYTES;
constexpr u32 TMO_OFF = CTRL_OFF + 256;
constexpr u32 TILED_OFF = CTRL_OFF + 512;
constexpr u32 RDY_OFF = CTRL_OFF + 2048;
constexpr u32 QUEUE_OFF = CTRL_OFF + 3072;
constexpr u32 PCNT_OFF = CTRL_OFF + 4096;
constexpr u32 NEED_OFF = CTRL_OFF + 5120;
constexpr u32 DUMMY_OFF = CTRL_OFF + 8192;
constexpr u32 FLAG_OFF = CTRL_OFF + 16384;
constexpr u32 FLAG_SRC = 80 * 128;
constexpr u32 WS_NEEDED = FLAG_OFF + NRANK * FLAG_SRC;

enum { K_X = 0, K_W = 1, K_P = 2, K_R = 3, K_O = 4 };
struct Lists { unsigned short v[2][NLIST]; unsigned short sh[NSH]; };
constexpr Lists make_lists() {
  Lists L{};
  bool shared[288] = {};
  for (int i = 0; i < NSH1; ++i) L.sh[i] = 96 + (i * 128 + 64) / NSH1;
  for (int i = 0; i < 64; ++i) L.sh[NSH1 + i] = 224 + i;
  for (int i = 0; i < NSH; ++i) shared[L.sh[i]] = true;
  for (int cr = 0; cr < 2; ++cr) {
    int n = 0, k = 0;
    if (cr) for (int i = 0; i < KP1; ++i) L.v[1][n++] = K_P << 12 | i;
    for (int w = 0; w < 288; ++w) {
      if (cr && w == LATEP) for (int i = KP1; i < KP; ++i) L.v[1][n++] = K_P << 12 | i;
      if (w < 48 && w % 6 == 0) for (int i = 0; i < 8; ++i) L.v[cr][n++] = K_X << 12 | (w / 6 * 8 + i);
      if (!cr && k < KP && w >= L.sh[4 * k]) { for (int j = 0; j < 4; ++j) L.v[0][n++] = K_R << 12 | (j * 64 + k); ++k; }
      if (!cr || !shared[w]) L.v[cr][n++] = K_W << 12 | w;
    }
    if (cr) for (int i = 0; i < 224; ++i) L.v[1][n++] = K_O << 12 | i;
  }
  return L;
}
constexpr Lists LISTS = make_lists();

struct Params {
  char* ws[NRANK];
  const int8_t* x;
  const int8_t* w;
  const float* sx;
  const float* sw;
  float* out;
  int rank;
};

#define FENCE() asm volatile("" ::: "memory")
#define WAIT_VM(n) do { FENCE(); __builtin_amdgcn_s_waitcnt(0x0F70 | (n)); FENCE(); } while (0)
#define LDS_LOAD(p) __hip_atomic_load((p), __ATOMIC_RELAXED, __HIP_MEMORY_SCOPE_WORKGROUP)
#define LDS_STORE(p, v) __hip_atomic_store((p), (v), __ATOMIC_RELAXED, __HIP_MEMORY_SCOPE_WORKGROUP)
#define LDS_ADD(p, v) __hip_atomic_fetch_add((p), (v), __ATOMIC_RELAXED, __HIP_MEMORY_SCOPE_WORKGROUP)
#define GPU_LOAD(p) __hip_atomic_load((p), __ATOMIC_RELAXED, __HIP_MEMORY_SCOPE_AGENT)
#define GPU_ADD(p, v) __hip_atomic_fetch_add((p), (v), __ATOMIC_RELAXED, __HIP_MEMORY_SCOPE_AGENT)
#define SYS_LOAD(p) __hip_atomic_load((p), __ATOMIC_RELAXED, __HIP_MEMORY_SCOPE_SYSTEM)
#define MAKE_RSRC(p, bytes) __builtin_amdgcn_make_buffer_rsrc((void*)(p), 0, (int)(bytes), 0x00020000)

template <class R>
__device__ __forceinline__ void store16(u32x4 d, R r, int voff, u32 soff) {
  __builtin_amdgcn_raw_buffer_store_b128(d, r, voff, soff, AUX_SYS);
  asm volatile("s_nop 1" ::"v"(d) : "memory");
}
template <class R>
__device__ __forceinline__ void store_block(const u32x4 (&d)[4], R r, int voff, u32 soff, int stride = 1024) {
#pragma unroll
  for (int j = 0; j < 4; ++j) __builtin_amdgcn_raw_buffer_store_b128(d[j], r, voff, soff + j * stride, AUX_SYS);
  asm volatile("s_nop 1" ::"v"(d[0]), "v"(d[1]), "v"(d[2]), "v"(d[3]) : "memory");
}

__device__ __forceinline__ void decode_need(int w, int& lg, int& c) {
  if (w < 192) { const int sp = w / 48, r = w - sp * 48; c = r / 6; lg = sp * 6 + r - c * 6; }
  else if (w < 224) { const int r = w - 192; c = r >> 2; lg = 24 + (r & 3); }
  else { const int r = w - 224; c = r >> 3; lg = 28 + (r & 7); }
}
__device__ __forceinline__ int global_group(int h, int lg) { return lg < 28 ? 28 * h + lg : 28 + lg; }

template <class R>
__device__ __forceinline__ void load_x_block(R rx, int gm, int c, int lane, u32x4 (&o)[4]) {
#pragma unroll
  for (int a = 0; a < 4; ++a)
    o[a] = __builtin_amdgcn_raw_buffer_load_b128(rx, (lane & 15) * KS + (lane >> 4) * 16, (gm * 64 + a * 16) * KS + c * 64, 0);
}
template <class R>
__device__ __forceinline__ void load_w_block(R rw, int grp, int c, int lane, u32 (&raw)[16]) {
#pragma unroll
  for (int i = 0; i < 16; ++i)
    raw[i] = __builtin_amdgcn_raw_buffer_load_b32(rw, (lane >> 4) * 16 * GN + (lane & 15) * 4, (c * 64 + i) * GN + grp * 64, 0);
}
__device__ __forceinline__ void tile_w_block(const u32 (&raw)[16], u32x4 (&o)[4]) {
#pragma unroll
  for (int k = 0; k < 4; ++k) {
    const u32 t0 = __builtin_amdgcn_perm(raw[4 * k + 1], raw[4 * k], 0x05010400u);
    const u32 t1 = __builtin_amdgcn_perm(raw[4 * k + 1], raw[4 * k], 0x07030602u);
    const u32 t2 = __builtin_amdgcn_perm(raw[4 * k + 3], raw[4 * k + 2], 0x05010400u);
    const u32 t3 = __builtin_amdgcn_perm(raw[4 * k + 3], raw[4 * k + 2], 0x07030602u);
    o[0][k] = __builtin_amdgcn_perm(t2, t0, 0x05040100u);
    o[1][k] = __builtin_amdgcn_perm(t2, t0, 0x07060302u);
    o[2][k] = __builtin_amdgcn_perm(t3, t1, 0x05040100u);
    o[3][k] = __builtin_amdgcn_perm(t3, t1, 0x07060302u);
  }
}

struct Smem {
  union {
    u64 tile[3][16][2][64];
    u32x4 table[NLIST + 1];
    char fill[96 * 1024];
  };
  u32 arrive[8];
  u32 bad;
  u32 pad[3];
};

__device__ __forceinline__ bool deadline_hit(u32 spins, u64& t0, u32* tmo, u32 call) {
  if (spins == 0) t0 = __builtin_amdgcn_s_memrealtime();
  if ((spins & 63) != 63) return false;
  if (GPU_LOAD(tmo) == call || __builtin_amdgcn_s_memrealtime() - t0 > DEADLINE_TICKS) {
    __hip_atomic_store(tmo, call, __ATOMIC_RELAXED, __HIP_MEMORY_SCOPE_AGENT);
    return true;
  }
  return false;
}

__device__ __forceinline__ u32 poll_ready(u32* fptr, u32 todo, int lane, u32* tmo, u32 call) {
  u64 t0 = 0;
  for (u32 spins = 0;; ++spins) {
    u32 f = call;
    if (lane < 16) f = SYS_LOAD(fptr);
    const u64 b = __ballot((int)(f - call) >= 0);
    const u32 ready = (u32)(b & (b >> 8)) & todo;
    if (ready) { FENCE(); return ready; }
    if (deadline_hit(spins, t0, tmo, call)) return 0;
    __builtin_amdgcn_s_sleep(4);
  }
}

__global__ void __launch_bounds__(NTHREADS) gemm_gather_kernel(Params P) {
  __shared__ __attribute__((aligned(16))) Smem sm;
  const int tid = threadIdx.x;
  const int lane = tid & 63;
  const int wave = __builtin_amdgcn_readfirstlane(tid >> 6);
  const int bid = blockIdx.x;
  const int rank = P.rank;
  const int h = rank & 1, pos = rank >> 1;
  char* myws = P.ws[rank];
  u32* ctr = (u32*)(myws + CTRL_OFF);
  u32* tmo = (u32*)(myws + TMO_OFF);
  u32* tiled = (u32*)(myws + TILED_OFF);
  const auto rx = MAKE_RSRC(P.x, GM * KS);
  const auto rw = MAKE_RSRC(P.w, KS * GN);
  const auto rl = MAKE_RSRC(myws, WS_NEEDED);
  const int lane0 = lane == 0 ? 0 : OOB;

  if (bid >= PULL0) {
    const u64 tstart = __builtin_amdgcn_s_memrealtime();
    const u32 cnt = GPU_LOAD(ctr);
    const int link = (bid - PULL0) >> 3;
    const int peer = (rank + 1 + link) & 7;
    const int cross = (peer ^ rank) & 1;
    const u32 dummy = DUMMY_OFF + (bid - PULL0) * 128;
    const u32 nlist = cross ? NL_CR : NL_IN;
    if (tid <= (int)NLIST) {
      u32x4 e = {0, 0, dummy, 0};
      if (tid < (int)nlist) {
        const u32 code = LISTS.v[cross][tid], kind = code >> 12, p = code & 4095;
        if (kind == K_X) {
          const int c = p >> 3, hr = p & 7;
          const int gm = (2 * (hr >> 1) + 1 - h) * 8 + 2 * pos + (hr & 1);
          e[0] = XST_OFF + (c * 64 + gm) * 4096;
          e[1] = peer * SRC_BYTES + (c * 16 + 8 + hr) * 4096;
          e[2] = FLAG_OFF + peer * FLAG_SRC + (64 + 8 + hr) * 128 + c * 4;
          e[3] = 1;
        } else if (kind == K_O) {
          const int lg = p >> 3, sub = (p >> 2) & 1, mi = p & 3;
          e[0] = OBX_OFF + (pos * 224 + p) * 4096;
          e[1] = (((2 * (peer >> 1) + sub) * 64 + mi * 16) * GN + (28 * (1 - h) + lg) * 64) * 4;
          e[3] = 1 | 2 | 8 | (p >> 2) << 4;
        } else {
          int w = p, s = peer;
          u32 meta = 1;
          if (kind != K_W) {
            w = LISTS.sh[4 * (p & 63) + (kind == K_P ? pos : peer >> 1)];
            const u32 late = (p & 63) >= KP1;
            if (kind == K_P) meta |= 4 | late << 4;
            else { s = 2 * (p >> 6) + 1 - h; meta = 1 | 8 | (56 + late * 4 + (p >> 6)) << 4; }
          }
          int lg, c;
          decode_need(w, lg, c);
          const int G = global_group(h, lg);
          e[0] = e[1] = s * SRC_BYTES + (128 + c * 64 + G) * 4096;
          e[2] = FLAG_OFF + s * FLAG_SRC + G * 128 + c * 4;
          e[3] = meta;
        }
      }
      sm.table[tid] = e;
    }
    const u32 call = (u32)__builtin_amdgcn_readfirstlane(cnt) / (u32)NWG + 1;
    __syncthreads();
    if (wave == 0) {
      while (__builtin_amdgcn_s_memrealtime() - tstart < PULL_GATE) __builtin_amdgcn_s_sleep(4);
      const auto rs = MAKE_RSRC(P.ws[peer], WS_NEEDED);
      const u32 pbase = (call & 1) ? PAR_BYTES : 0;
      bool failed = false;
      {
        u32* rdy = (u32*)(myws + RDY_OFF) + peer * 32;
        u64 t0 = 0;
        for (u32 spins = 0; (int)(SYS_LOAD(rdy) - call) < 0; ++spins) {
          if (deadline_hit(spins, t0, tmo, call)) { failed = true; break; }
          __builtin_amdgcn_s_sleep(2);
        }
        FENCE();
      }
      if (lane == 0) GPU_ADD(ctr, 1u);
      struct Ent { u32 so, dof, fo, meta; };
      auto lookup = [&](u32 i) __attribute__((always_inline)) -> Ent {
        const u32x4 e = sm.table[i < NLIST ? i : NLIST];
        return Ent{(u32)__builtin_amdgcn_readfirstlane(e[0]), (u32)__builtin_amdgcn_readfirstlane(e[1]),
                   (u32)__builtin_amdgcn_readfirstlane(e[2]), (u32)__builtin_amdgcn_readfirstlane(e[3])};
      };
      auto fetch = [&](const Ent& e, u32x4 (&d)[4]) __attribute__((always_inline)) {
        const int vo = (e.meta & 1) ? lane * 16 : OOB;
#pragma unroll
        for (int j = 0; j < 4; ++j) d[j] = __builtin_amdgcn_raw_buffer_load_b128(rs, vo + j * 1024, pbase + e.so, AUX_SYS);
        FENCE();
      };
      const int ovoff = (((lane >> 4) * 4) * GN + (lane & 15) * 4) * 4;
      auto put = [&](const Ent& e, u32x4 (&d)[4]) __attribute__((always_inline)) {
        const bool o = e.meta & 2;
        const u32 nanm = failed ? 0x7FC00000u : 0u;
#pragma unroll
        for (int j = 0; j < 4; ++j) d[j] |= u32x4{nanm, nanm, nanm, nanm};
        const auto rd = MAKE_RSRC(o ? (void*)P.out : (void*)myws, o ? MS * GN * 4 : WS_NEEDED);
        const int vo = (e.meta & 1) ? (o ? ovoff : lane * 16) : OOB;
        store_block(d, rd, vo, o ? e.dof : pbase + e.dof, o ? GN * 4 : 1024);
      };
      auto flag = [&](const Ent& e) __attribute__((always_inline)) {
        __builtin_amdgcn_raw_buffer_store_b32(call, rl, lane0, failed ? dummy : e.fo, AUX_SYS);
        FENCE();
        if ((e.meta & 4) && lane == 0) GPU_ADD((u32*)(myws + PCNT_OFF) + link * 32 + (e.meta >> 4) * 16, 1u);
        FENCE();
      };
      auto ticket = [&](bool v) __attribute__((always_inline)) -> u32 {
        const u32 tk = (u32)__builtin_amdgcn_raw_ptr_buffer_atomic_add_i32(1, rl, v ? lane0 : OOB,
                                                                           QUEUE_OFF + ((call & 1) * 8 + link) * 64, 0);
        FENCE();
        return tk;
      };
      auto peek = [&]() __attribute__((always_inline)) -> u32 {
        const u32 f = __builtin_amdgcn_raw_buffer_load_b32(rl, lane * 4, NEED_OFF + peer * 256, AUX_SYS);
        FENCE();
        return f;
      };
      u32x4 d0[4], d1[4];
      Ent e0 = lookup(bid & 7), e1 = lookup((bid & 7) + 8), e2 = lookup((bid & 7) + 16), e3 = lookup((bid & 7) + 24);
      fetch(e0, d0);
      fetch(e1, d1);
      WAIT_VM(4);  put(e0, d0); u32 tk0 = ticket(true), rd0 = peek(); fetch(e2, d0);
      WAIT_VM(10); put(e1, d1); u32 tk1 = ticket(true), rd1 = peek(); fetch(e3, d1);
      u64 known = failed ? ~0ull : 0, tw = 0;
      u32 p0 = NLIST, p1 = NLIST, idle = 0;
      bool has0 = true, has1 = true, ended = false, full = false;
#define PULL_STEP(D, TK, RD, HAS, OHAS, CNT) {                                                           \
        if (!full) WAIT_VM(CNT); else WAIT_VM(0);                                                        \
        if (HAS) {                                                                                       \
          const u32 i = (u32)__builtin_amdgcn_readfirstlane(TK) + 32;                                    \
          if (i >= nlist) ended = true; else if (p0 >= NLIST) p0 = i; else p1 = i;                       \
        }                                                                                                \
        Ent e4 = lookup(p0);                                                                             \
        known |= __ballot((int)(RD - call) >= 0);                                                        \
        put(e2, D);                                                                                      \
        if (!(e4.meta & 8) || ((known >> (e4.meta >> 4)) & 1)) { p0 = p1; p1 = NLIST; }                  \
        else {                                                                                           \
          e4.meta = 0; e4.fo = dummy;                                                                    \
          if (deadline_hit(idle++, tw, tmo, call)) { failed = true; known = ~0ull; }                     \
        }                                                                                                \
        HAS = !ended && (p0 < NLIST) + (p1 < NLIST) + OHAS < 2;                                          \
        TK = ticket(HAS);                                                                                \
        RD = peek();                                                                                     \
        fetch(e4, D);                                                                                    \
        flag(e0);                                                                                        \
        if (full) { if (e1.meta & 1) flag(e1); e1.meta = 0; e1.fo = dummy; }                             \
        full = !((e2.meta & e4.meta & 1) && HAS);                                                        \
        if (e2.meta & 2) e2.meta = 0;                             \
        e0 = e1; e1 = e2; e2 = e3; e3 = e4;                                                              \
      }
      PULL_STEP(d0, tk0, rd0, has0, has1, 10)
      PULL_STEP(d1, tk1, rd1, has1, has0, 11)
#pragma unroll 1
      do {
        PULL_STEP(d0, tk0, rd0, has0, has1, 12)
        PULL_STEP(d1, tk1, rd1, has1, has0, 12)
      } while (((e0.meta | e1.meta | e2.meta | e3.meta) & 1) || has0 || has1 || p0 < NLIST);
#undef PULL_STEP
    } else if (wave == 2 && (bid & 7) == 0) {
      __builtin_amdgcn_raw_buffer_store_b32(0, rl, lane0, QUEUE_OFF + ((~call & 1) * 8 + link) * 64, AUX_SYS);
    } else if (wave == 3 && (bid & 7) == 1 && cross) {
      while (__builtin_amdgcn_s_memrealtime() - tstart < PULL_GATE + 100) __builtin_amdgcn_s_sleep(16);
      for (int late = 0; late < 2; ++late) {
        u32* pc = (u32*)(myws + PCNT_OFF) + link * 32 + late * 16;
        const u32 target = call * (late ? KP - KP1 : KP1);
        u64 t0 = 0;
        bool ok = true;
        for (u32 spins = 0; (int)(GPU_LOAD(pc) - target) < 0; ++spins) {
          if (deadline_hit(spins, t0, tmo, call)) { ok = false; break; }
          if (late) __builtin_amdgcn_s_sleep(32); else __builtin_amdgcn_s_sleep(8);
        }
        FENCE();
        if (ok && GPU_LOAD(tmo) != call) {
#pragma unroll
          for (int u = 1; u < 4; ++u) {
            const auto ro = MAKE_RSRC(P.ws[(rank + 2 * u) & 7], WS_NEEDED);
            __builtin_amdgcn_raw_buffer_store_b32(call, ro, lane0, NEED_OFF + rank * 256 + (56 + late * 4 + (peer >> 1)) * 4, AUX_SYS);
          }
        }
      }
    } else if (wave == 1 && bid == PULL0) {
      const u32 target = call * (NTILE / 8);
      u64 t0 = 0;
      bool ok = true;
      for (u32 spins = 0;; ++spins) {
        u32 f = target;
        if (lane < 8) f = GPU_LOAD(tiled + lane * 32);
        if (__all((int)(f - target) >= 0)) break;
        if (deadline_hit(spins, t0, tmo, call)) { ok = false; break; }
        __builtin_amdgcn_s_sleep(1);
      }
      FENCE();
      if (ok) {
#pragma unroll
        for (int i = 1; i < NRANK; ++i) {
          const auto rs = MAKE_RSRC(P.ws[(rank + i) & 7], WS_NEEDED);
          __builtin_amdgcn_raw_buffer_store_b32(call, rs, lane0, RDY_OFF + rank * 128, AUX_SYS);
        }
      }
    }
    return;
  }

  const int q = bid < NXP ? NTILE + bid - 8 : bid - NXP;
  const int xcd = bid & 7;
  const int idx = q >> 3;
  const int src = (rank - wave) & 7;
  auto init_lds = [&]() __attribute__((always_inline)) {
    u64* z = &sm.tile[0][0][0][0];
#pragma unroll
    for (int i = 0; i < 3 * 16 * 2 * 64 / NTHREADS; ++i) z[i * NTHREADS + tid] = 0;
    if (tid < 8) sm.arrive[tid] = 0;
    if (tid == 8) sm.bad = 0;
    asm volatile("s_waitcnt lgkmcnt(0)" ::: "memory");
    __builtin_amdgcn_s_barrier();
  };
  u32 call;
  if (bid < NXP && wave < 4) {
    const int peer = (rank + 1 + (bid >> 3)) & 7;
    const int slot = (bid & 7) * 4 + wave;
    u32x4 b0[4], b1[4];
    load_x_block(rx, peer * 8 + (slot & 7), slot >> 3, lane, b0);
    const u32 cnt = GPU_LOAD(ctr);
    load_x_block(rx, peer * 8 + (slot & 7), 4 + (slot >> 3), lane, b1);
    FENCE();
    call = (u32)__builtin_amdgcn_readfirstlane(cnt) / (u32)NWG + 1;
    const auto rs = MAKE_RSRC(P.ws[peer], WS_NEEDED);
    const u32 doff = ((call & 1) ? PAR_BYTES : 0) + (u32)rank * SRC_BYTES + (u32)((slot >> 3) * 16 + (slot & 7)) * 4096;
    const u32 foff = FLAG_OFF + (u32)rank * FLAG_SRC + (u32)(64 + (slot & 7)) * 128 + (u32)(slot >> 3) * 4;
    store_block(b0, rs, lane * 16, doff);
    init_lds();
    WAIT_VM(0);
    __builtin_amdgcn_raw_buffer_store_b32(call, rs, lane0, foff, AUX_SYS);
    store_block(b1, rs, lane * 16, doff + 64 * 4096);
    WAIT_VM(0);
    __builtin_amdgcn_raw_buffer_store_b32(call, rs, lane0, foff + 16, AUX_SYS);
    FENCE();
  } else if (bid >= NXP) {
    const int n = wave * NTILE + q;
    const bool isw = n < 512;
    const int a = n & 63, c = (n >> 6) & 7;
    u32x4 o[4];
    u32 raw[16];
    if (isw) load_w_block(rw, a, c, lane, raw);
    else load_x_block(rx, a, c, lane, o);
    const u32 cnt = GPU_LOAD(ctr);
    FENCE();
    init_lds();
    call = (u32)__builtin_amdgcn_readfirstlane(cnt) / (u32)NWG + 1;
    const u32 pbase = (call & 1) ? PAR_BYTES : 0;
    const u32 dbase = pbase + (u32)rank * SRC_BYTES;
    int frow = -1;
    if (isw) {
      tile_w_block(raw, o);
      store_block(o, rl, lane * 16, dbase + (u32)(128 + n) * 4096);
      frow = a;
    } else if (n < 1024) {
      const int owner = a >> 3, mt = a & 7;
      const bool mine = owner == rank, helped = ((owner ^ rank) & 1) && (mt >> 1) == pos;
      if (!helped) store_block(o, rl, lane * 16, pbase + XST_OFF + (u32)(n - 512) * 4096);
      if (mine || helped) {
        const int rt = mine ? mt : 8 + (owner >> 1) * 2 + (mt & 1);
        store_block(o, rl, lane * 16, dbase + (u32)(c * 16 + rt) * 4096);
        frow = 64 + rt;
      }
    }
    WAIT_VM(0);
    if (lane == 0 && LDS_ADD(&sm.arrive[3], 1u) == 7) GPU_ADD(tiled + xcd * 32, 1u);
    if (frow >= 0)
      __builtin_amdgcn_raw_buffer_store_b32(call, rl, lane0, FLAG_OFF + (u32)rank * FLAG_SRC + (u32)frow * 128 + (u32)c * 4, AUX_SYS);
    FENCE();
  } else {
    const u32 cnt = GPU_LOAD(ctr);
    FENCE();
    init_lds();
    call = (u32)__builtin_amdgcn_readfirstlane(cnt) / (u32)NWG + 1;
  }
  if (bid < 8) {
    __syncthreads();
    if (tid == 0) GPU_ADD(ctr, 1u);
    return;
  }

  const float scale = P.sx[0] * P.sw[0];
  const u32 pbase = (call & 1) ? PAR_BYTES : 0;
  const u32 sbase = pbase + (u32)src * SRC_BYTES;
  u32* fsrc = (u32*)(myws + FLAG_OFF + (u32)src * FLAG_SRC);
  const int nphase = idx < 16 ? 3 : 2;

#pragma unroll 1
  for (int t = 0; t < nphase; ++t) {
    const bool tail = t == 2 && idx >= 8;
    const int lg = tail ? 28 + xcd : t * 12 + 2 * (idx >> 2) + (xcd >> 2);
    const int rt = tail ? idx - 8 : (xcd & 3) * 4 + (idx & 3);
    const int G = global_group(h, lg);
    i32x4 acc[4][4];
#pragma unroll
    for (int a = 0; a < 4; ++a)
#pragma unroll
      for (int b = 0; b < 4; ++b) acc[a][b] = i32x4{0, 0, 0, 0};
    u32* fptr = fsrc + (lane < 8 ? G : 64 + rt) * 32 + (lane & 7);
    u32 todo = 0xFF, ready = 0;
    u32x4 af[2][4], bf[2][4];
#pragma unroll
    for (int n = 0; n <= NCH; ++n) {
      if (n < NCH) {
        if (!ready) {
          ready = poll_ready(fptr, todo, lane, tmo, call);
          if (!ready) {
            LDS_STORE(&sm.bad, 1u);
            ready = todo;
          }
        }
        const int c = __builtin_ctz(ready);
        ready &= ready - 1;
        todo &= ~(1u << c);
        const u32 xo = sbase + (u32)(c * 16 + rt) * 4096;
        const u32 wo = sbase + (u32)(128 + c * 64 + G) * 4096;
#pragma unroll
        for (int j = 0; j < 4; ++j) bf[n & 1][j] = __builtin_amdgcn_raw_buffer_load_b128(rl, lane * 16 + j * 1024, wo, AUX_SYS);
#pragma unroll
        for (int a = 0; a < 4; ++a) af[n & 1][a] = __builtin_amdgcn_raw_buffer_load_b128(rl, lane * 16 + a * 1024, xo, AUX_SYS);
        __builtin_amdgcn_sched_barrier(0);
      }
      if (n > 0) {
#pragma unroll
        for (int a = 0; a < 4; ++a)
#pragma unroll
          for (int j = 0; j < 4; ++j)
            acc[a][j] = __builtin_amdgcn_mfma_i32_16x16x64_i8((i32x4)af[(n - 1) & 1][a], (i32x4)bf[(n - 1) & 1][j], acc[a][j], 0, 0, 0);
        __builtin_amdgcn_sched_barrier(0);
      }
    }
    int ln = lane;
    asm volatile("" : "+v"(ln));
#pragma unroll
    for (int a = 0; a < 4; ++a)
#pragma unroll
      for (int i = 0; i < 4; ++i)
#pragma unroll
        for (int jp = 0; jp < 2; ++jp) {
          const int lo = acc[a][2 * jp][i], hi = acc[a][2 * jp + 1][i];
          LDS_ADD(&sm.tile[t][a * 4 + i][jp][ln], ((u64)(u32)(hi + (lo >> 31)) << 32) | (u32)lo);
        }
    int e0, e1;
    const bool last = t + 1 == nphase;
    if (last) {
      __syncthreads();
      e0 = wave * 4; e1 = wave < 4 ? e0 + 4 : e0;
    } else {
      asm volatile("s_waitcnt lgkmcnt(0)" ::: "memory");
      u32 tk = 0;
      if (ln == 0) tk = LDS_ADD(&sm.arrive[t], 1u);
      tk = __builtin_amdgcn_readfirstlane(tk);
      FENCE();
      e0 = 0; e1 = tk == 7 ? 16 : 0;
      if (t == 0 && tk == 7 && ln == 0) GPU_ADD(ctr, 1u);
    }
    if (e0 < e1) {
      const bool bad = LDS_LOAD(&sm.bad) != 0;
      const bool own = rt < 8;
      const int op = (rt - 8) >> 1;
      const auto ro = MAKE_RSRC(own ? (void*)P.out : (void*)myws, own ? MS * GN * 4 : WS_NEEDED);
      const int voff = own ? (((ln >> 4) * 4) * GN + (ln & 15) * 4) * 4 : ln * 16;
      const u32 obx = pbase + OBX_OFF + (u32)(op * 224 + (lg * 2 + (rt & 1)) * 4) * 4096;
      for (int e = e0; e < e1; e += 4) {
        u64 v[4][2];
#pragma unroll
        for (int k = 0; k < 4; ++k) { v[k][0] = sm.tile[t][e + k][0][ln]; v[k][1] = sm.tile[t][e + k][1][ln]; }
#pragma unroll
        for (int k = 0; k < 4; ++k) {
          const int s0 = (int)(u32)v[k][0], s2 = (int)(u32)v[k][1];
          const int s1 = (int)(u32)(v[k][0] >> 32) - (s0 >> 31), s3 = (int)(u32)(v[k][1] >> 32) - (s2 >> 31);
          f32x4 y = f32x4{(float)s0 * scale, (float)s1 * scale, (float)s2 * scale, (float)s3 * scale};
          if (bad) y = f32x4{__builtin_nanf(""), __builtin_nanf(""), __builtin_nanf(""), __builtin_nanf("")};
          const u32 so = own ? (u32)((rt * 64 + e * 4 + k) * GN + G * 64) * 4 : obx + (u32)(e + k) * 1024;
          store16(__builtin_bit_cast(u32x4, y), ro, voff, so);
        }
      }
      if (!own) {
        WAIT_VM(0);
        u32 n = 3;
        if (last && ln == 0) n = LDS_ADD(&sm.arrive[4], 1u);
        if (__builtin_amdgcn_readfirstlane(n) == 3) {
          const auto rs = MAKE_RSRC(P.ws[(2 * op + 1 - h) & 7], WS_NEEDED);
          __builtin_amdgcn_raw_buffer_store_b32(call, rs, lane0, NEED_OFF + rank * 256 + (lg * 2 + (rt & 1)) * 4, AUX_SYS);
        }
      }
    }
  }
}

}

extern "C" void kernel_launch(int rank, int nranks, ncclComm_t comm,
                              hipStream_t stream,
                              void* const* d_in, const int* in_sizes, int n_in,
                              void* d_out, int out_size,
                              const int* in_dtypes, int out_dtype,
                              void* const* d_out_all,
                              void* const* d_ws_all, size_t ws_bytes) {
  if (nranks != NRANK || n_in != 4 || ws_bytes < WS_NEEDED || out_size != MS * GN ||
      in_sizes[0] != GM * KS || in_sizes[1] != KS * GN || (unsigned)rank >= (unsigned)NRANK) {
    fprintf(stderr, "kernel_launch: unexpected problem shape\n");
    return;
  }
  Params p;
  for (int i = 0; i < NRANK; ++i) p.ws[i] = (char*)d_ws_all[i];
  p.x = (const int8_t*)d_in[0];
  p.w = (const int8_t*)d_in[1];
  p.sx = (const float*)d_in[2];
  p.sw = (const float*)d_in[3];
  p.out = (float*)d_out;
  p.rank = rank;
  hipLaunchKernelGGL(gemm_gather_kernel, dim3(NWG), dim3(NTHREADS), 0, stream, p);
  HIP_CHECK(hipGetLastError());
}
```
